```python
import numpy as np

kernel_name = "gpu_a2a_gemm_m4096_k4096_n4096_e4m3e5m2_dequant"


ABSMAX_RTOL = 0.05

COMPUTE_DTYPE = "float32"
EPILOGUE = "dequant"
IN_QUANT = None
OUT_QUANT = None
IN_FP8 = "e4m3e5m2"
IN_DTYPES = {"x": "e4m3", "w_mat": "e5m2",
             "scale_x": "float32", "scale_w": "float32"}
WORLD_SIZE = 8

def _snap_fp8(x, mant_bits, max_abs, min_normal):
    a = np.abs(x.astype(np.float64))
    _m, _e = np.frexp(a)
    q = np.where(a >= min_normal, np.ldexp(1.0, _e - 1 - mant_bits),
                 min_normal * 2.0 ** -mant_bits)
    snapped = np.minimum(np.round(a / q) * q, max_abs)
    return (np.sign(x) * snapped).astype(np.float32)

def setup_inputs():

    x = _snap_fp8(np.random.randn(4096, 4096).astype(np.float32), 3, 448.0, 2.0 ** -6)
    w_mat = _snap_fp8(np.random.randn(4096, 4096).astype(np.float32), 2, 57344.0, 2.0 ** -14)
    scale_x = (0.02 * (1.0 + np.random.rand(1))).astype(np.float32)
    scale_w = (0.02 * (1.0 + np.random.rand(1))).astype(np.float32)
    return {"x": x, "w_mat": w_mat, "scale_x": scale_x, "scale_w": scale_w}

def reference(x, w_mat, scale_x, scale_w):
    acc = (x @ w_mat).astype(np.float32)
    y = (acc * (scale_x * scale_w)).astype(np.float32)
    return y
```

```cpp
#include <hip/hip_runtime.h>
#include <rccl/rccl.h>
#include <cstdio>
#include <cstdint>

typedef int v2i __attribute__((ext_vector_type(2)));
typedef int v4i __attribute__((ext_vector_type(4)));
typedef int v16i __attribute__((ext_vector_type(16)));
typedef float v4f __attribute__((ext_vector_type(4)));
typedef unsigned long long u64;
#define LAS __attribute__((address_space(3)))
#define CAS __attribute__((address_space(4)))

constexpr int NR = 8;
constexpr int KD = 4096, ND = 4096;
constexpr int MS = 512;
constexpr int RING = 40960;
constexpr int LDS_BYTES = 4 * RING;
constexpr unsigned RECV_BYTES = 2u << 20;
constexpr unsigned CNT_OFF = 3 * RECV_BYTES;
constexpr unsigned TMO_OFF = CNT_OFF + 1024 * 64;
constexpr unsigned WS_NEED = TMO_OFF + 128;
constexpr unsigned STAMP_OFF = 8u << 20;
constexpr unsigned OOB = 0x80000000u;
constexpr u64 DEADLINE = 40000000ull;

struct KArgs {
  uint8_t* ws[NR];
  const uint8_t* x; const uint8_t* w; uint8_t* own; float* out; const float* sx; const float* sw;
  int rank; int pad[3];
};

__device__ __forceinline__ u64 rt() { u64 t; asm volatile("s_memrealtime %0\n\ts_waitcnt lgkmcnt(0)" : "=s"(t) :: "memory"); return t; }
__device__ __forceinline__ unsigned rfl(unsigned v) { return __builtin_amdgcn_readfirstlane(v); }
__device__ __forceinline__ unsigned min4(unsigned m, v4i a) {
  m = min(min(m, (unsigned)a.x), (unsigned)a.y);
  return min(min(m, (unsigned)a.z), (unsigned)a.w);
}

#define STAMP_END() do {} while (0)
#define STAMP(i) do {} while (0)
#define SB() __builtin_amdgcn_sched_barrier(0)
#define SGB(n) do { __builtin_amdgcn_sched_group_barrier(0x8, 1, 0); __builtin_amdgcn_sched_group_barrier(0x2, n, 0); } while (0)

__global__ void __launch_bounds__(256, 1) a2a_gemm(KArgs p) {
  extern __shared__ __attribute__((aligned(16))) uint8_t smem[];
  LAS uint8_t* lds = (LAS uint8_t*)smem;
  const int b = blockIdx.x, g = b & 7, gi = b >> 3;
  const CAS uint8_t* kbase = (const CAS uint8_t*)__builtin_amdgcn_kernarg_segment_ptr();
  const v16i ka = *(const volatile CAS v16i*)(kbase + 64);
  const v2i kp = *(const volatile CAS v2i*)(kbase + g * 8);
  const int wave = rfl(threadIdx.x >> 6), lane = threadIdx.x & 63;
  unsigned seqv, seq, rbuf;
  unsigned voffA = lane * 16;
  unsigned voffX = (lane >> 3) * MS + (((lane & 7) ^ (lane >> 3)) * 16);
  const unsigned wbase = wave * RING;
  unsigned xrd = wbase + (lane & 15) * 128 + (((lane >> 4) ^ (lane & 7)) * 16);
  unsigned cnto = CNT_OFF + (b * 4 + wave) * 64;
#define PTR(lo, hi) ((uint8_t*)(((u64)(unsigned)(hi) << 32) | (unsigned)(lo)))
#define GOTARGS() \
  asm volatile("" :: "s"(ka), "s"(kp)); \
  const uint8_t* px = PTR(ka[0], ka[1]); const uint8_t* pw = PTR(ka[2], ka[3]); uint8_t* own = PTR(ka[4], ka[5]); \
  const int rank = ka[12]; \
  const __amdgpu_buffer_rsrc_t srdX = __builtin_amdgcn_make_buffer_rsrc((void*)px, (short)0, KD * MS, 0x00020000); \
  const __amdgpu_buffer_rsrc_t srdW = __builtin_amdgcn_make_buffer_rsrc((void*)pw, (short)0, KD * ND, 0x00020000); \
  const __amdgpu_buffer_rsrc_t srdA = __builtin_amdgcn_make_buffer_rsrc((void*)own, (short)0, WS_NEED, 0x00020000); \
  STAMP(0)
#define LDSEQ() asm volatile("buffer_load_dword %0, off, %1, %2 sc0 sc1" : "=&v"(seqv) : "s"(srdA), "s"(cnto) : "memory")
#define GOTSEQ() do { seq = rfl(seqv) + 1; rbuf = (seq % 3) * RECV_BYTES; asm volatile("s_nop 4" : "+s"(rbuf)); } while (0)
#define STSEQ() asm volatile("buffer_store_dword %0, off, %1, %2 sc1" :: "v"(seq), "s"(srdA), "s"(cnto) : "memory")
#define DMA(ldsaddr, vo, srd, so) asm volatile("s_mov_b32 m0, %0\n\ts_nop 0\n\tbuffer_load_dwordx4 %1, %2, %3 offen lds" \
      :: "s"(ldsaddr), "v"(vo), "s"(srd), "s"(so) : "memory")
#define NTOFF 0x7420
#define NTILE(c) ((c) * 8 + ((g + (NTOFF >> (4 * (c)))) & 7))
#define NPUSH 4

  if (gi < 4) {
    if (gi < NPUSH && g != 7) {
      const int q = 6 - g;
      const int first = (q < 4 ? 5 * q : 4 * q + 4) + gi * 4 + wave;
      asm volatile("" : "+v"(voffA), "+v"(voffX), "+v"(xrd));
      GOTARGS();
      const __amdgpu_buffer_rsrc_t srdP = __builtin_amdgcn_make_buffer_rsrc((void*)PTR(kp[0], kp[1]), (short)0, 3 * RECV_BYTES, 0x00020000);
      const unsigned xb = (unsigned)(((rank + 1 + g) & 7) * MS) * MS;
      const unsigned sb = (unsigned)rank * 262144;
#define PBLK(i) ((first + 4 * NPUSH * (i)) & 31)
#define PQ(i) ((4 * NPUSH * (i)) >> 5)
#define PLD(i, h) DMA(wbase + ((i) % 20) * 2048 + (h) * 1024, voffX, srdX, xb + PQ(i) * 128 + (PBLK(i) * 16 + (h) * 8) * MS)
#define ENC(e) do { e.x = e.x ? e.x : 0x80; e.y = e.y ? e.y : 0x80; e.z = e.z ? e.z : 0x80; e.w = e.w ? e.w : 0x80; } while (0)
#define PST(i) do { v4i e0 = *(LAS v4i*)(lds + xrd + ((i) % 20) * 2048), e1 = *(LAS v4i*)(lds + (xrd ^ 64) + ((i) % 20) * 2048); ENC(e0); ENC(e1); \
      asm volatile("buffer_store_dwordx4 %0, %2, %3, %4 offen sc0 sc1\n\tbuffer_store_dwordx4 %1, %2, %3, %4 offen offset:1024 sc0 sc1\n\ts_nop 1" \
      :: "v"(e0), "v"(e1), "v"(voffA), "s"(srdP), "s"(rbuf + sb + PQ(i) * 65536 + PBLK(i) * 2048) : "memory"); } while (0)
      PLD(0, 0);
      PLD(0, 1);
      LDSEQ();
      PLD(1, 0);
      PLD(1, 1);
      PLD(2, 0);
      PLD(2, 1);
      asm volatile("s_waitcnt vmcnt(4)" : "+v"(seqv) :: "memory");
      GOTSEQ();
      STAMP(1);
      PST(0);
      PLD(3, 0);
      PLD(3, 1);
      STAMP(2);
      asm volatile("s_waitcnt vmcnt(6)" ::: "memory");
      PST(1);
      PLD(4, 0);
      PLD(4, 1);
      STAMP(3);
      asm volatile("s_waitcnt vmcnt(8)" ::: "memory");
      PST(2);
      PLD(5, 0);
      PLD(5, 1);
      STAMP(4);
      asm volatile("s_waitcnt vmcnt(8)" ::: "memory");
      PST(3);
      PLD(6, 0);
      PLD(6, 1);
      STAMP(5);
      asm volatile("s_waitcnt vmcnt(8)" ::: "memory");
      PST(4);
      PLD(7, 0);
      PLD(7, 1);
      STAMP(6);
      asm volatile("s_waitcnt vmcnt(8)" ::: "memory");
      PST(5);
      STAMP(7);
      asm volatile("s_waitcnt vmcnt(6)" ::: "memory");
      PST(6);
      STAMP(8);
      asm volatile("s_waitcnt vmcnt(4)" ::: "memory");
      PST(7);
      STAMP(9);
      __builtin_amdgcn_s_barrier();
      const unsigned zb = ((seq + 2) % 3) * RECV_BYTES + ((rank + 1 + g) & 7) * 262144 + (gi * 4 + wave) * (65536 / NPUSH);
#pragma unroll 4
      for (int i = 0; i < 64 / NPUSH; ++i) __builtin_amdgcn_raw_buffer_store_b128((v4i){0, 0, 0, 0}, srdA, voffA, zb + i * 1024, 17);
      STAMP(10);
      STSEQ();
      STAMP_END();
    } else {
      const int nf = g == 7 ? 16 : 4 * (4 - NPUSH), f = (g == 7 ? gi : gi - NPUSH) * 4 + wave;
      unsigned voffF = (lane >> 5) * ND + NTILE((lane >> 3) & 3) * 128 + (lane & 7) * 16;
      asm volatile("" : "+v"(voffF));
      GOTARGS();
#define FETCH0 256
#pragma unroll 1
      for (int m = f; m < (512 - FETCH0) / 2 * 7; m += nf) {
        const int rp = m / 7, qq = m - rp * 7;
        DMA(wbase, voffF, srdW, (unsigned)(((rank + 1 + qq) & 7) * MS + FETCH0 + rp * 2) * ND);
        asm volatile("s_waitcnt vmcnt(8)" ::: "memory");
      }
      STAMP(10);
      STAMP_END();
    }
  } else {
    const int idx = gi - 4, r = idx >> 2, c = idx & 3;
    const int nt = NTILE(c);
    const int blk0 = r < 4 ? 5 * r : 4 * r + 4;
    unsigned voffW0 = (lane >> 3) * ND + (((lane & 7) ^ (lane >> 4)) * 16);
    unsigned voffW1 = voffW0 ^ 64;
    unsigned lb[8];
#pragma unroll
    for (int t = 0; t < 8; ++t)
      lb[t] = wbase + (lane >> 4) * 1024 + ((lane & 15) >> 1) * 128 + (((t) ^ (((lane & 15) >> 2) | (((lane >> 4) & 1) << 2))) * 16) + (lane & 1) * 8;
    unsigned sq[8], sj[8];
    sq[0] = 0; sj[0] = wave;
#pragma unroll
    for (int s = 1; s < 8; ++s) {
      const int n = 4 * (s - 1) + wave, j = (n * 37) >> 8;
      int q = r - (n - 7 * j); q += q < 0 ? 7 : 0;
      sq[s] = q + 1; sj[s] = j;
    }
    asm volatile("" : "+v"(voffA), "+v"(voffX), "+v"(xrd), "+v"(voffW0), "+v"(voffW1), "+v"(lb[0]), "+v"(lb[1]), "+v"(lb[2]), "+v"(lb[3]),
                 "+v"(lb[4]), "+v"(lb[5]), "+v"(lb[6]), "+v"(lb[7]));
    asm volatile("" : "+s"(sq[1]), "+s"(sq[2]), "+s"(sj[2]));
    GOTARGS();
    float* pout = (float*)PTR(ka[6], ka[7]);
    const float* psx = (const float*)PTR(ka[8], ka[9]); const float* psw = (const float*)PTR(ka[10], ka[11]);
#define SK(s) ((((rank + sq[s]) & 7) * 4 + sj[s]))
#define WK(s) (SK(s) * (128 * ND) + nt * 128)
    unsigned* tmo = (unsigned*)(own + TMO_OFF);
    const unsigned xloc = (unsigned)(rank * MS + blk0 * 16) * MS + wave * 128;
    v4f acc[5][8];
    v4i a[3][10];
    v2i bf[8][4], bh[8][4], bg[8][4];
    unsigned ao0[3], ao2[3], ao4[3], bad = 0, am;
    const unsigned no5 = r < 4 ? 0u : 1u;
    float scale = *(const CAS float*)(uintptr_t)psx * *(const CAS float*)(uintptr_t)psw;
#define DMAW(ldsoff, vo, kb, rowoff) DMA(wbase + (ldsoff), vo, srdW, (kb) + (rowoff))
#define DMAC(ldsoff, vo, kb, rowoff) do { unsigned so_; asm volatile("s_add_u32 m0, %1, %5\n\ts_add_u32 %0, %4, %6\n\tbuffer_load_dwordx4 %2, %3, %0 offen lds" \
      : "=&s"(so_) : "s"(wbase), "v"(vo), "s"(srdW), "s"(kb), "n"(ldsoff), "n"(rowoff) : "memory", "scc"); } while (0)
#define ABASE(s) (rbuf + SK(s) * 65536)
#define SETA(s) do { constexpr int k_ = (s) % 3; ao0[k_] = ABASE(s) + blk0 * 2048; ao2[k_] = ao0[k_] + 4096; ao4[k_] = r < 4 ? ao0[k_] + 8192 : OOB; \
      if ((s) == 1) asm volatile("" : "+v"(scale)); } while (0)
#define DMAW0(j, h) do { const unsigned pr_ = ((j) + r) & 7; \
      DMAW(pr_ * 2048 + (h) * 1024, ((h) ? voffW1 : voffW0), WK(0), ((pr_ >> 2) * 64 + (pr_ & 1) * 32 + (h) * 16 + ((pr_ >> 1) & 1) * 8) * ND); } while (0)
#define DMAR(s, u0, j, g4) do { const unsigned v_ = ((j) + wave) & 3, p_ = (u0) + v_ - ((u0) + v_ >= 10 ? 10 : 0); \
      DMAW(p_ * 4096 + (g4) * 1024, (((g4) & 1) ? voffW1 : voffW0), WK(s), ((v_ >> 1) * 64 + (g4) * 16 + (v_ & 1) * 8) * ND); } while (0)
#define WARM(i) DMAW(16384 + ((3 + wave) & 3) * 4096, voffW0, (unsigned)((((rank + 1 + r) & 7) * 4 + 1 + ((i) >> 2)) * 128 + wave * 32 + ((i) & 3) * 8) * ND, NTILE((c + 1) & 3) * 128)
#define LDA(dstv, so, imm) asm volatile("buffer_load_dwordx4 %0, %1, %2, %3 offen offset:" #imm " sc0 sc1" : "=&v"(dstv) : "v"(voffA), "s"(srdA), "s"(so) : "memory")
#define XPOS(blk) ((blk) < 4 ? 32768 + (blk) * 2048 : ((7 + r) & 7) * 2048)
#define DMAX(e) do { unsigned e_ = (e) + ((0x8530 >> (4 * c)) & 15); e_ -= e_ >= 10 ? 10 : 0; \
      DMA(wbase + XPOS(e_ >> 1) + (e_ & 1) * 1024, voffX, srdX, (e_ < 8 || r < 4) ? xloc + e_ * 8 * MS : OOB); } while (0)
#define RDX(dstv, blk, u) dstv = *(LAS v4i*)(lds + ((xrd ^ ((u) * 64)) + XPOS(blk)))
#define TR(base, imm) __builtin_amdgcn_ds_read_tr8_b64_v2i32((LAS v2i*)(lds + (base) + (imm)))
#define TRA(d, base, imm) asm volatile("ds_read_b64_tr_b8 %0, %1 offset:%2" : "=&a"(d) : "v"(base), "n"(imm) : "memory")
#define BGV(t, h) do { asm volatile("" : "+a"(bg[t][2 * (h)]), "+a"(bg[t][2 * (h) + 1])); asm volatile("" : "+v"(bg[t][2 * (h)]), "+v"(bg[t][2 * (h) + 1])); } while (0)
#define CAT4(bb) __builtin_shufflevector(__builtin_shufflevector(bb[0], bb[1], 0, 1, 2, 3), __builtin_shufflevector(bb[2], bb[3], 0, 1, 2, 3), 0, 1, 2, 3, 4, 5, 6, 7)
#define CAT2(a0, a1) __builtin_shufflevector(a0, a1, 0, 1, 2, 3, 4, 5, 6, 7)
#define MMA0(pad, c, bb, a0, a1) asm volatile(pad "v_mfma_f32_16x16x128_f8f6f4 %0, %1, %2, 0 cbsz:1" : "=&a"(c) : "v"(CAT4(bb)), "v"(CAT2(a0, a1)))
#define MMA(pad, c, bb, a0, a1) asm volatile(pad "v_mfma_f32_16x16x128_f8f6f4 %0, %1, %2, %0 cbsz:1" : "+a"(c) : "v"(CAT4(bb)), "v"(CAT2(a0, a1)))
#define NOP "s_nop 1\n\t"
#define MINBLK(B, k) min4(min4(0xffffffffu, a[B][2 * (k)]), a[B][2 * (k) + 1])
#define MISSING(expr) (__builtin_amdgcn_ballot_w64((expr) == 0u) != 0)
#define SPIN(...) do { \
      if (__builtin_expect(MISSING(am), 0)) { \
        unsigned sp_ = 0; const u64 t0 = rt(); \
        do { \
          if ((++sp_ & 255) == 0 || bad) { \
            unsigned tv_; asm volatile("global_load_dword %0, %1, off sc1\n\ts_waitcnt vmcnt(0)" : "=&v"(tv_) : "v"(tmo) : "memory"); \
            if (bad || rfl(tv_) == seq || rt() - t0 > DEADLINE) { \
              asm volatile("global_store_dword %0, %1, off sc1\n\ts_waitcnt vmcnt(0)" :: "v"(tmo), "v"(seq) : "memory"); \
              bad = 1; break; } } \
          __VA_ARGS__ \
        } while (MISSING(am)); \
      } } while (0)
    unsigned lwr = wbase + (lane >> 4) * 256 + (((lane & 15) ^ ((lane >> 4) * 4)) * 16);
#define PREG(i) ((((i) + 1) % 5) * 8192)
#define PUT(i, t) asm volatile("ds_write_b128 %0, %1 offset:%2" :: "v"(lwr), "a"(acc[i][t]), "n"(PREG(i) + (t) * 1024) : "memory")
    const __amdgpu_buffer_rsrc_t srdO = __builtin_amdgcn_make_buffer_rsrc((void*)pout, (short)0, MS * ND * 4, 0x00020000);
    unsigned voffO = ((lane >> 3) * ND + (lane & 7) * 4) * 4;
    const unsigned ob = (unsigned)(blk0 * 16 * ND + nt * 128 + wave * 32) * 4;
    const unsigned lrd = (2 * wave + ((lane >> 2) & 1)) * 1024 + (lane & 3) * 256 + (((lane >> 3) ^ ((lane & 3) * 4)) * 16);
    unsigned lg[2][3] = {{lrd, lrd + 65536, lrd + 131072}, {lrd ^ 128, (lrd ^ 128) + 65536, (lrd ^ 128) + 131072}};
    v4f gv[2][4];
#define OUT_AUX "sc1"
#define PIN() asm volatile("" : "+v"(lwr), "+v"(voffO), "+v"(lg[0][0]), "+v"(lg[0][1]), "+v"(lg[0][2]), "+v"(lg[1][0]), "+v"(lg[1][1]), "+v"(lg[1][2]))
#define BAR(n) do { asm volatile("s_waitcnt lgkmcnt(" #n ")" ::: "memory"); __builtin_amdgcn_s_barrier(); asm volatile("" ::: "memory"); } while (0)
#define GETOFF(i, j) (PREG(i) + (j) * RING)
#define GET(i, h, j) asm volatile("ds_read_b128 %0, %1 offset:%2" : "=&v"(gv[h][j]) : "v"(lg[h][GETOFF(i, j) >> 16]), "n"(GETOFF(i, j) & 65535) : "memory")
#define ADD4(h, x, y) do { _Pragma("unroll") for (int e_ = 0; e_ < 4; ++e_) { float r_; asm volatile("v_add_f32 %0, %1, %2" : "=v"(r_) : "v"(gv[h][x][e_]), "v"(gv[h][y][e_])); gv[h][x][e_] = r_; } } while (0)
#define MUL4(h) do { _Pragma("unroll") for (int e_ = 0; e_ < 4; ++e_) { float r_; asm volatile("v_mul_f32 %0, %1, %2" : "=v"(r_) : "v"(gv[h][0][e_]), "v"(scale)); gv[h][0][e_] = r_; } } while (0)
#define OUT(i, h) asm volatile("buffer_store_dwordx4 %0, %1, %2, %3 offen " OUT_AUX "\n\ts_nop 1" :: "v"(gv[h][0]), "v"(voffO), "s"(srdO), \
      "s"(((i) < 4 || r < 4) ? ob + ((i) * 16 + (h) * 8) * ND * 4 : OOB) : "memory")
#define POISON(i) do { if (__builtin_expect(bad, 0)) { const float nan_ = __builtin_nanf(""); \
      _Pragma("unroll 1") for (int t_ = 0; t_ < 8; ++t_) *(LAS v4f*)(lds + lwr + PREG(i) + t_ * 1024) = (v4f){nan_, nan_, nan_, nan_}; \
      asm volatile("s_waitcnt lgkmcnt(0)" ::: "memory"); } } while (0)
    DMAW0(0, 0);
    DMAW0(0, 1);
    DMAW0(1, 0);
    DMAW0(1, 1);
    DMAX(0);
    DMAX(1);
    DMAX(2);
    LDSEQ();
    DMAR(1, 4, 0, 0);
    DMAR(1, 4, 0, 1);
    DMAR(1, 4, 0, 2);
    DMAR(1, 4, 0, 3);
    WARM(0);
    WARM(1);
    WARM(2);
    WARM(3);
    STAMP(1);
    DMAX(3);
    DMAX(4);
    DMAX(5);
    DMAX(6);
    DMAX(7);
    DMAX(8);
    DMAX(9);
    DMAW0(2, 0);
    DMAW0(2, 1);
    DMAW0(3, 0);
    DMAW0(3, 1);
    DMAW0(4, 0);
    DMAW0(4, 1);
    DMAW0(5, 0);
    DMAW0(5, 1);
    DMAW0(6, 0);
    DMAW0(6, 1);
    asm volatile("s_waitcnt vmcnt(10)" ::: "memory");
    RDX(a[0][0], 0, 0);
    RDX(a[0][1], 0, 1);
    RDX(a[0][2], 1, 0);
    RDX(a[0][3], 1, 1);
    RDX(a[0][4], 2, 0);
    RDX(a[0][5], 2, 1);
    RDX(a[0][6], 3, 0);
    RDX(a[0][7], 3, 1);
    RDX(a[0][8], 4, 0);
    RDX(a[0][9], 4, 1);
    asm volatile("s_waitcnt lgkmcnt(0)" ::: "memory");
    DMAW0(7, 0);
    DMAW0(7, 1);
    PIN();
    asm volatile("s_waitcnt vmcnt(0)" ::: "memory");
    STAMP(2);
    bf[0][0] = TR(lb[0], 0);
    bf[0][1] = TR(lb[0], 4096);
    bf[0][2] = TR(lb[0], 8192);
    bf[0][3] = TR(lb[0], 12288);
    bf[1][0] = TR(lb[1], 0);
    bf[1][1] = TR(lb[1], 4096);
    bf[1][2] = TR(lb[1], 8192);
    bf[1][3] = TR(lb[1], 12288);
    bf[2][0] = TR(lb[2], 0);
    bf[2][1] = TR(lb[2], 4096);
    bf[2][2] = TR(lb[2], 8192);
    bf[2][3] = TR(lb[2], 12288);
    bf[3][0] = TR(lb[3], 0);
    bf[3][1] = TR(lb[3], 4096);
    bf[3][2] = TR(lb[3], 8192);
    bf[3][3] = TR(lb[3], 12288);
    bf[4][0] = TR(lb[4], 0);
    bf[4][1] = TR(lb[4], 4096);
    bf[4][2] = TR(lb[4], 8192);
    bf[4][3] = TR(lb[4], 12288);
    bf[5][0] = TR(lb[5], 0);
    bf[5][1] = TR(lb[5], 4096);
    bf[5][2] = TR(lb[5], 8192);
    bf[5][3] = TR(lb[5], 12288);
    bf[6][0] = TR(lb[6], 0);
    bf[6][1] = TR(lb[6], 4096);
    bf[6][2] = TR(lb[6], 8192);
    bf[6][3] = TR(lb[6], 12288);
    bf[7][0] = TR(lb[7], 0);
    bf[7][1] = TR(lb[7], 4096);
    bf[7][2] = TR(lb[7], 8192);
    bf[7][3] = TR(lb[7], 12288);
    asm volatile("s_waitcnt lgkmcnt(2)" ::: "memory");
    DMAR(7, 8, 0, 0);
    DMAR(7, 8, 0, 1);
    DMAR(7, 8, 0, 2);
    DMAR(7, 8, 0, 3);
    WARM(4);
    WARM(5);
    WARM(6);
    WARM(7);
    MMA0("", acc[0][0], bf[0], a[0][0], a[0][1]);
    SB();
    DMAR(1, 4, 1, 0);
    SB();
    DMAR(1, 4, 1, 1);
    SB();
    MMA0("", acc[1][0], bf[0], a[0][2], a[0][3]);
    SB();
    DMAR(1, 4, 1, 2);
    SB();
    DMAR(1, 4, 1, 3);
    SB();
    MMA0("", acc[2][0], bf[0], a[0][4], a[0][5]);
    SB();
    DMAR(1, 4, 2, 0);
    SB();
    DMAR(1, 4, 2, 1);
    SB();
    MMA0("", acc[3][0], bf[0], a[0][6], a[0][7]);
    SB();
    DMAR(1, 4, 2, 2);
    SB();
    DMAR(1, 4, 2, 3);
    SB();
    MMA0("", acc[4][0], bf[0], a[0][8], a[0][9]);
    SB();
    DMAR(1, 4, 3, 0);
    SB();
    DMAR(1, 4, 3, 1);
    SB();
    MMA0("", acc[0][1], bf[1], a[0][0], a[0][1]);
    SB();
    DMAR(1, 4, 3, 2);
    SB();
    DMAR(1, 4, 3, 3);
    SB();
    MMA0("", acc[1][1], bf[1], a[0][2], a[0][3]);
    SB();
    DMAR(7, 8, 1, 0);
    SB();
    MMA0("", acc[2][1], bf[1], a[0][4], a[0][5]);
    SB();
    DMAR(7, 8, 1, 1);
    SB();
    DMAR(7, 8, 1, 2);
    SB();
    MMA0("", acc[3][1], bf[1], a[0][6], a[0][7]);
    SB();
    DMAR(7, 8, 1, 3);
    SB();
    DMAR(7, 8, 2, 0);
    SB();
    MMA0("", acc[4][1], bf[1], a[0][8], a[0][9]);
    SB();
    DMAR(7, 8, 2, 1);
    SB();
    DMAR(7, 8, 2, 2);
    SB();
    MMA0("", acc[0][2], bf[2], a[0][0], a[0][1]);
    SB();
    DMAR(7, 8, 2, 3);
    SB();
    MMA0("", acc[1][2], bf[2], a[0][2], a[0][3]);
    SB();
    DMAR(7, 8, 3, 0);
    SB();
    DMAR(7, 8, 3, 1);
    SB();
    MMA0("", acc[2][2], bf[2], a[0][4], a[0][5]);
    SB();
    DMAR(7, 8, 3, 2);
    SB();
    DMAR(7, 8, 3, 3);
    SB();
    MMA0("", acc[3][2], bf[2], a[0][6], a[0][7]);
    SB();
    asm volatile("" : "+v"(seqv) :: "memory");
    GOTSEQ();
    SETA(1);
    LDA(a[1][0], ao0[1], 0);
    SB();
    LDA(a[1][1], ao0[1], 1024);
    SB();
    MMA0("", acc[4][2], bf[2], a[0][8], a[0][9]);
    SB();
    asm volatile("s_waitcnt vmcnt(14)" ::: "memory");
    bh[0][0] = TR(lb[0], 16384);
    bh[1][0] = TR(lb[1], 16384);
    bh[2][0] = TR(lb[2], 16384);
    bh[3][0] = TR(lb[3], 16384);
    SB();
    LDA(a[1][2], ao0[1], 2048);
    SB();
    MMA0("", acc[0][3], bf[3], a[0][0], a[0][1]);
    SB();
    bh[4][0] = TR(lb[4], 16384);
    bh[5][0] = TR(lb[5], 16384);
    bh[6][0] = TR(lb[6], 16384);
    bh[7][0] = TR(lb[7], 16384);
    SB();
    LDA(a[1][3], ao0[1], 3072);
    SB();
    MMA0("", acc[1][3], bf[3], a[0][2], a[0][3]);
    SB();
    bh[0][1] = TR(lb[0], 20480);
    bh[1][1] = TR(lb[1], 20480);
    bh[2][1] = TR(lb[2], 20480);
    bh[3][1] = TR(lb[3], 20480);
    SB();
    LDA(a[1][4], ao2[1], 0);
    SB();
    LDA(a[1][5], ao2[1], 1024);
    SB();
    MMA0("", acc[2][3], bf[3], a[0][4], a[0][5]);
    SB();
    bh[4][1] = TR(lb[4], 20480);
    bh[5][1] = TR(lb[5], 20480);
    bh[6][1] = TR(lb[6], 20480);
    bh[7][1] = TR(lb[7], 20480);
    SB();
    LDA(a[1][6], ao2[1], 2048);
    SB();
    MMA0("", acc[3][3], bf[3], a[0][6], a[0][7]);
    SB();
    bh[0][2] = TR(lb[0], 24576);
    bh[1][2] = TR(lb[1], 24576);
    bh[2][2] = TR(lb[2], 24576);
    bh[3][2] = TR(lb[3], 24576);
    SB();
    LDA(a[1][7], ao2[1], 3072);
    SB();
    LDA(a[1][8], ao4[1], 0);
    SB();
    MMA0("", acc[4][3], bf[3], a[0][8], a[0][9]);
    SB();
    bh[4][2] = TR(lb[4], 24576);
    bh[5][2] = TR(lb[5], 24576);
    bh[6][2] = TR(lb[6], 24576);
    bh[7][2] = TR(lb[7], 24576);
    SB();
    LDA(a[1][9], ao4[1], 1024);
    SB();
    MMA0("", acc[0][4], bf[4], a[0][0], a[0][1]);
    SB();
    bh[0][3] = TR(lb[0], 28672);
    bh[1][3] = TR(lb[1], 28672);
    bh[2][3] = TR(lb[2], 28672);
    bh[3][3] = TR(lb[3], 28672);
    SB();
    asm volatile("s_waitcnt lgkmcnt(15)" ::: "memory");
    DMAC(8192, voffW0, WK(2), 0);
    SB();
    MMA0("", acc[1][4], bf[4], a[0][2], a[0][3]);
    SB();
    bh[4][3] = TR(lb[4], 28672);
    bh[5][3] = TR(lb[5], 28672);
    bh[6][3] = TR(lb[6], 28672);
    bh[7][3] = TR(lb[7], 28672);
    SB();
    DMAC(9216, voffW1, WK(2), 65536);
    SB();
    DMAC(10240, voffW0, WK(2), 131072);
    SB();
    MMA0("", acc[2][4], bf[4], a[0][4], a[0][5]);
    SB();
    DMAC(11264, voffW1, WK(2), 196608);
    SB();
    MMA0("", acc[3][4], bf[4], a[0][6], a[0][7]);
    SB();
    asm volatile("s_waitcnt lgkmcnt(15)" ::: "memory");
    DMAC(12288, voffW0, WK(2), 32768);
    SB();
    DMAC(13312, voffW1, WK(2), 98304);
    SB();
    MMA0("", acc[4][4], bf[4], a[0][8], a[0][9]);
    SB();
    asm volatile("s_waitcnt vmcnt(16)" ::: "memory");
    TRA(bg[0][0], lb[0], 32768);
    TRA(bg[1][0], lb[1], 32768);
    TRA(bg[2][0], lb[2], 32768);
    TRA(bg[3][0], lb[3], 32768);
    SB();
    DMAC(14336, voffW0, WK(2), 163840);
    SB();
    MMA0("", acc[0][5], bf[5], a[0][0], a[0][1]);
    SB();
    TRA(bg[4][0], lb[4], 32768);
    TRA(bg[5][0], lb[5], 32768);
    TRA(bg[6][0], lb[6], 32768);
    TRA(bg[7][0], lb[7], 32768);
    SB();
    DMAC(15360, voffW1, WK(2), 229376);
    SB();
    MMA0("", acc[1][5], bf[5], a[0][2], a[0][3]);
    SB();
    TRA(bg[0][1], lb[0], 36864);
    TRA(bg[1][1], lb[1], 36864);
    TRA(bg[2][1], lb[2], 36864);
    TRA(bg[3][1], lb[3], 36864);
    SB();
    asm volatile("s_waitcnt lgkmcnt(15)" ::: "memory");
    DMAC(16384, voffW0, WK(2), 262144);
    SB();
    DMAC(17408, voffW1, WK(2), 327680);
    SB();
    MMA0("", acc[2][5], bf[5], a[0][4], a[0][5]);
    SB();
    TRA(bg[4][1], lb[4], 36864);
    TRA(bg[5][1], lb[5], 36864);
    TRA(bg[6][1], lb[6], 36864);
    TRA(bg[7][1], lb[7], 36864);
    SB();
    DMAC(18432, voffW0, WK(2), 393216);
    SB();
    MMA0("", acc[3][5], bf[5], a[0][6], a[0][7]);
    SB();
    TRA(bg[0][2], lb[0], 0);
    TRA(bg[1][2], lb[1], 0);
    TRA(bg[2][2], lb[2], 0);
    TRA(bg[3][2], lb[3], 0);
    SB();
    DMAC(19456, voffW1, WK(2), 458752);
    SB();
    asm volatile("s_waitcnt lgkmcnt(15)" ::: "memory");
    DMAC(20480, voffW0, WK(2), 294912);
    SB();
    MMA0("", acc[4][5], bf[5], a[0][8], a[0][9]);
    SB();
    TRA(bg[4][2], lb[4], 0);
    TRA(bg[5][2], lb[5], 0);
    TRA(bg[6][2], lb[6], 0);
    TRA(bg[7][2], lb[7], 0);
    SB();
    DMAC(21504, voffW1, WK(2), 360448);
    SB();
    MMA0("", acc[0][6], bf[6], a[0][0], a[0][1]);
    SB();
    TRA(bg[0][3], lb[0], 4096);
    TRA(bg[1][3], lb[1], 4096);
    TRA(bg[2][3], lb[2], 4096);
    TRA(bg[3][3], lb[3], 4096);
    SB();
    DMAC(22528, voffW0, WK(2), 425984);
    SB();
    MMA0("", acc[1][6], bf[6], a[0][2], a[0][3]);
    SB();
    TRA(bg[4][3], lb[4], 4096);
    TRA(bg[5][3], lb[5], 4096);
    TRA(bg[6][3], lb[6], 4096);
    TRA(bg[7][3], lb[7], 4096);
    SB();
    DMAC(23552, voffW1, WK(2), 491520);
    SB();
    asm volatile("s_waitcnt lgkmcnt(15)" ::: "memory");
    DMAC(24576, voffW0, WK(3), 0);
    SB();
    MMA0("", acc[2][6], bf[6], a[0][4], a[0][5]);
    SB();
    asm volatile("s_waitcnt vmcnt(25)" : "+v"(a[1][0]), "+v"(a[1][1]) :: "memory");
    am = MINBLK(1, 0);
    asm volatile("" : "+v"(am));
    SB();
    DMAC(25600, voffW1, WK(3), 65536);
    SB();
    MMA0("", acc[3][6], bf[6], a[0][6], a[0][7]);
    SB();
    asm volatile("s_waitcnt vmcnt(24)" : "+v"(a[1][2]), "+v"(a[1][3]) :: "memory");
    am = min(am, MINBLK(1, 1));
    asm volatile("" : "+v"(am));
    SB();
    DMAC(26624, voffW0, WK(3), 131072);
    SB();
    DMAC(27648, voffW1, WK(3), 196608);
    SB();
    MMA0("", acc[4][6], bf[6], a[0][8], a[0][9]);
    SB();
    asm volatile("s_waitcnt vmcnt(24)" : "+v"(a[1][4]), "+v"(a[1][5]) :: "memory");
    am = min(am, MINBLK(1, 2));
    asm volatile("" : "+v"(am));
    SB();
    asm volatile("s_waitcnt lgkmcnt(15)" ::: "memory");
    DMAC(28672, voffW0, WK(3), 32768);
    SB();
    MMA0("", acc[0][7], bf[7], a[0][0], a[0][1]);
    SB();
    asm volatile("s_waitcnt vmcnt(23)" : "+v"(a[1][6]), "+v"(a[1][7]) :: "memory");
    am = min(am, MINBLK(1, 3));
    asm volatile("" : "+v"(am));
    SB();
    DMAC(29696, voffW1, WK(3), 98304);
    SB();
    MMA0("", acc[1][7], bf[7], a[0][2], a[0][3]);
    SB();
    asm volatile("s_waitcnt vmcnt(22)" : "+v"(a[1][8]), "+v"(a[1][9]) :: "memory");
    am = min(am, (MINBLK(1, 4) | no5));
    asm volatile("" : "+v"(am));
    SB();
    DMAC(30720, voffW0, WK(3), 163840);
    SB();
    DMAC(31744, voffW1, WK(3), 229376);
    SB();
    MMA0("", acc[2][7], bf[7], a[0][4], a[0][5]);
    SB();
    SPIN(LDA(a[1][0], ao0[1], 0); LDA(a[1][1], ao0[1], 1024); LDA(a[1][2], ao0[1], 2048); LDA(a[1][3], ao0[1], 3072); LDA(a[1][4], ao2[1], 0); LDA(a[1][5], ao2[1], 1024); LDA(a[1][6], ao2[1], 2048); LDA(a[1][7], ao2[1], 3072); LDA(a[1][8], ao4[1], 0); LDA(a[1][9], ao4[1], 1024); asm volatile("s_waitcnt vmcnt(0)" : "+v"(a[1][0]), "+v"(a[1][1]), "+v"(a[1][2]), "+v"(a[1][3]), "+v"(a[1][4]), "+v"(a[1][5]), "+v"(a[1][6]), "+v"(a[1][7]), "+v"(a[1][8]), "+v"(a[1][9]) :: "memory"); am = min(min(min(min(MINBLK(1, 0), MINBLK(1, 1)), MINBLK(1, 2)), MINBLK(1, 3)), (MINBLK(1, 4) | no5)););
    SB();
    asm volatile("s_waitcnt lgkmcnt(15)" ::: "memory");
    DMAC(32768, voffW0, WK(3), 262144);
    SB();
    MMA0("", acc[3][7], bf[7], a[0][6], a[0][7]);
    SB();
    DMAC(33792, voffW1, WK(3), 327680);
    SB();
    DMAC(34816, voffW0, WK(3), 393216);
    SB();
    MMA0("", acc[4][7], bf[7], a[0][8], a[0][9]);
    SB();
    DMAC(35840, voffW1, WK(3), 458752);
    SB();
    STAMP(3);
    MMA("", acc[0][0], bh[0], a[1][0], a[1][1]);
    SB();
    asm volatile("s_waitcnt vmcnt(24)" ::: "memory");
    bf[0][0] = TR(lb[0], 8192);
    SB();
    asm volatile("s_waitcnt lgkmcnt(15)" ::: "memory");
    DMAC(36864, voffW0, WK(3), 294912);
    SB();
    MMA("", acc[1][0], bh[0], a[1][2], a[1][3]);
    SB();
    bf[1][0] = TR(lb[1], 8192);
    SB();
    MMA("", acc[2][0], bh[0], a[1][4], a[1][5]);
    SB();
    bf[2][0] = TR(lb[2], 8192);
    SB();
    DMAC(37888, voffW1, WK(3), 360448);
    SB();
    MMA("", acc[3][0], bh[0], a[1][6], a[1][7]);
    SB();
    bf[3][0] = TR(lb[3], 8192);
    SB();
    MMA("", acc[4][0], bh[0], a[1][8], a[1][9]);
    SB();
    bf[4][0] = TR(lb[4], 8192);
    SB();
    DMAC(38912, voffW0, WK(3), 425984);
    SB();
    MMA("", acc[0][1], bh[1], a[1][0], a[1][1]);
    SB();
    bf[5][0] = TR(lb[5], 8192);
    SB();
    MMA("", acc[1][1], bh[1], a[1][2], a[1][3]);
    SB();
    bf[6][0] = TR(lb[6], 8192);
    SB();
    SETA(2);
    LDA(a[2][0], ao0[2], 0);
    SB();
    MMA("", acc[2][1], bh[1], a[1][4], a[1][5]);
    SB();
    bf[7][0] = TR(lb[7], 8192);
    SB();
    MMA("", acc[3][1], bh[1], a[1][6], a[1][7]);
    SB();
    asm volatile("s_waitcnt vmcnt(24)" ::: "memory");
    bf[0][1] = TR(lb[0], 12288);
    SB();
    LDA(a[2][1], ao0[2], 1024);
    SB();
    MMA("", acc[4][1], bh[1], a[1][8], a[1][9]);
    SB();
    bf[1][1] = TR(lb[1], 12288);
    SB();
    MMA("", acc[0][2], bh[2], a[1][0], a[1][1]);
    SB();
    bf[2][1] = TR(lb[2], 12288);
    SB();
    LDA(a[2][2], ao0[2], 2048);
    SB();
    MMA("", acc[1][2], bh[2], a[1][2], a[1][3]);
    SB();
    bf[3][1] = TR(lb[3], 12288);
    SB();
    LDA(a[2][3], ao0[2], 3072);
    SB();
    MMA("", acc[2][2], bh[2], a[1][4], a[1][5]);
    SB();
    bf[4][1] = TR(lb[4], 12288);
    SB();
    MMA("", acc[3][2], bh[2], a[1][6], a[1][7]);
    SB();
    bf[5][1] = TR(lb[5], 12288);
    SB();
    LDA(a[2][4], ao2[2], 0);
    SB();
    MMA("", acc[4][2], bh[2], a[1][8], a[1][9]);
    SB();
    bf[6][1] = TR(lb[6], 12288);
    SB();
    MMA("", acc[0][3], bh[3], a[1][0], a[1][1]);
    SB();
    bf[7][1] = TR(lb[7], 12288);
    SB();
    LDA(a[2][5], ao2[2], 1024);
    SB();
    MMA("", acc[1][3], bh[3], a[1][2], a[1][3]);
    SB();
    asm volatile("s_waitcnt vmcnt(25)" ::: "memory");
    bf[0][2] = TR(lb[0], 16384);
    SB();
    MMA("", acc[2][3], bh[3], a[1][4], a[1][5]);
    SB();
    bf[1][2] = TR(lb[1], 16384);
    SB();
    LDA(a[2][6], ao2[2], 2048);
    SB();
    MMA("", acc[3][3], bh[3], a[1][6], a[1][7]);
    SB();
    bf[2][2] = TR(lb[2], 16384);
    SB();
    MMA("", acc[4][3], bh[3], a[1][8], a[1][9]);
    SB();
    bf[3][2] = TR(lb[3], 16384);
    SB();
    LDA(a[2][7], ao2[2], 3072);
    SB();
    MMA("", acc[0][4], bh[4], a[1][0], a[1][1]);
    SB();
    bf[4][2] = TR(lb[4], 16384);
    SB();
    MMA("", acc[1][4], bh[4], a[1][2], a[1][3]);
    SB();
    bf[5][2] = TR(lb[5], 16384);
    SB();
    LDA(a[2][8], ao4[2], 0);
    SB();
    MMA("", acc[2][4], bh[4], a[1][4], a[1][5]);
    SB();
    bf[6][2] = TR(lb[6], 16384);
    SB();
    MMA("", acc[3][4], bh[4], a[1][6], a[1][7]);
    SB();
    bf[7][2] = TR(lb[7], 16384);
    SB();
    LDA(a[2][9], ao4[2], 1024);
    SB();
    MMA("", acc[4][4], bh[4], a[1][8], a[1][9]);
    SB();
    asm volatile("s_waitcnt vmcnt(25)" ::: "memory");
    bf[0][3] = TR(lb[0], 20480);
    SB();
    MMA("", acc[0][5], bh[5], a[1][0], a[1][1]);
    SB();
    bf[1][3] = TR(lb[1], 20480);
    SB();
    DMAC(39936, voffW1, WK(3), 491520);
    SB();
    MMA("", acc[1][5], bh[5], a[1][2], a[1][3]);
    SB();
    bf[2][3] = TR(lb[2], 20480);
    SB();
    asm volatile("s_waitcnt lgkmcnt(15)" ::: "memory");
    DMAC(0, voffW0, WK(4), 0);
    SB();
    MMA("", acc[2][5], bh[5], a[1][4], a[1][5]);
    SB();
    bf[3][3] = TR(lb[3], 20480);
    SB();
    MMA("", acc[3][5], bh[5], a[1][6], a[1][7]);
    SB();
    bf[4][3] = TR(lb[4], 20480);
    SB();
    DMAC(1024, voffW1, WK(4), 65536);
    SB();
    MMA("", acc[4][5], bh[5], a[1][8], a[1][9]);
    SB();
    bf[5][3] = TR(lb[5], 20480);
    SB();
    MMA("", acc[0][6], bh[6], a[1][0], a[1][1]);
    SB();
    bf[6][3] = TR(lb[6], 20480);
    SB();
    DMAC(2048, voffW0, WK(4), 131072);
    SB();
    MMA("", acc[1][6], bh[6], a[1][2], a[1][3]);
    SB();
    bf[7][3] = TR(lb[7], 20480);
    SB();
    MMA("", acc[2][6], bh[6], a[1][4], a[1][5]);
    SB();
    asm volatile("s_waitcnt vmcnt(12)" : "+v"(a[2][0]), "+v"(a[2][1]) :: "memory");
    am = MINBLK(2, 0);
    asm volatile("" : "+v"(am));
    SB();
    MMA("", acc[3][6], bh[6], a[1][6], a[1][7]);
    SB();
    asm volatile("s_waitcnt vmcnt(10)" : "+v"(a[2][2]), "+v"(a[2][3]) :: "memory");
    am = min(am, MINBLK(2, 1));
    asm volatile("" : "+v"(am));
    SB();
    MMA("", acc[4][6], bh[6], a[1][8], a[1][9]);
    SB();
    asm volatile("s_waitcnt vmcnt(8)" : "+v"(a[2][4]), "+v"(a[2][5]) :: "memory");
    am = min(am, MINBLK(2, 2));
    asm volatile("" : "+v"(am));
    SB();
    MMA("", acc[0][7], bh[7], a[1][0], a[1][1]);
    SB();
    asm volatile("s_waitcnt vmcnt(6)" : "+v"(a[2][6]), "+v"(a[2][7]) :: "memory");
    am = min(am, MINBLK(2, 3));
    asm volatile("" : "+v"(am));
    SB();
    MMA("", acc[1][7], bh[7], a[1][2], a[1][3]);
    SB();
    asm volatile("s_waitcnt vmcnt(4)" : "+v"(a[2][8]), "+v"(a[2][9]) :: "memory");
    am = min(am, (MINBLK(2, 4) | no5));
    asm volatile("" : "+v"(am));
    SB();
    MMA("", acc[2][7], bh[7], a[1][4], a[1][5]);
    SB();
    SPIN(LDA(a[2][0], ao0[2], 0); LDA(a[2][1], ao0[2], 1024); LDA(a[2][2], ao0[2], 2048); LDA(a[2][3], ao0[2], 3072); LDA(a[2][4], ao2[2], 0); LDA(a[2][5], ao2[2], 1024); LDA(a[2][6], ao2[2], 2048); LDA(a[2][7], ao2[2], 3072); LDA(a[2][8], ao4[2], 0); LDA(a[2][9], ao4[2], 1024); asm volatile("s_waitcnt vmcnt(0)" : "+v"(a[2][0]), "+v"(a[2][1]), "+v"(a[2][2]), "+v"(a[2][3]), "+v"(a[2][4]), "+v"(a[2][5]), "+v"(a[2][6]), "+v"(a[2][7]), "+v"(a[2][8]), "+v"(a[2][9]) :: "memory"); am = min(min(min(min(MINBLK(2, 0), MINBLK(2, 1)), MINBLK(2, 2)), MINBLK(2, 3)), (MINBLK(2, 4) | no5)););
    SB();
    MMA("", acc[3][7], bh[7], a[1][6], a[1][7]);
    SB();
    DMAC(3072, voffW1, WK(4), 196608);
    SB();
    MMA("", acc[4][7], bh[7], a[1][8], a[1][9]);
    SB();
    asm volatile("s_waitcnt lgkmcnt(15)" ::: "memory");
    DMAC(4096, voffW0, WK(4), 32768);
    SB();
    STAMP(4);
    MMA("", acc[0][0], bf[0], a[2][0], a[2][1]);
    SB();
    bh[0][0] = TR(lb[0], 24576);
    SB();
    DMAC(5120, voffW1, WK(4), 98304);
    SB();
    MMA("", acc[1][0], bf[0], a[2][2], a[2][3]);
    SB();
    bh[1][0] = TR(lb[1], 24576);
    SB();
    DMAC(6144, voffW0, WK(4), 163840);
    SB();
    MMA("", acc[2][0], bf[0], a[2][4], a[2][5]);
    SB();
    bh[2][0] = TR(lb[2], 24576);
    SB();
    DMAC(7168, voffW1, WK(4), 229376);
    SB();
    MMA("", acc[3][0], bf[0], a[2][6], a[2][7]);
    SB();
    bh[3][0] = TR(lb[3], 24576);
    SB();
    asm volatile("s_waitcnt lgkmcnt(15)" ::: "memory");
    DMAC(8192, voffW0, WK(4), 262144);
    SB();
    MMA("", acc[4][0], bf[0], a[2][8], a[2][9]);
    SB();
    bh[4][0] = TR(lb[4], 24576);
    SB();
    DMAC(9216, voffW1, WK(4), 327680);
    SB();
    MMA("", acc[0][1], bf[1], a[2][0], a[2][1]);
    SB();
    bh[5][0] = TR(lb[5], 24576);
    SB();
    DMAC(10240, voffW0, WK(4), 393216);
    SB();
    MMA("", acc[1][1], bf[1], a[2][2], a[2][3]);
    SB();
    bh[6][0] = TR(lb[6], 24576);
    SB();
    MMA("", acc[2][1], bf[1], a[2][4], a[2][5]);
    SB();
    bh[7][0] = TR(lb[7], 24576);
    SB();
    SETA(3);
    LDA(a[0][0], ao0[0], 0);
    SB();
    MMA("", acc[3][1], bf[1], a[2][6], a[2][7]);
    SB();
    bh[0][1] = TR(lb[0], 28672);
    SB();
    LDA(a[0][1], ao0[0], 1024);
    SB();
    MMA("", acc[4][1], bf[1], a[2][8], a[2][9]);
    SB();
    bh[1][1] = TR(lb[1], 28672);
    SB();
    MMA("", acc[0][2], bf[2], a[2][0], a[2][1]);
    SB();
    bh[2][1] = TR(lb[2], 28672);
    SB();
    LDA(a[0][2], ao0[0], 2048);
    SB();
    MMA("", acc[1][2], bf[2], a[2][2], a[2][3]);
    SB();
    bh[3][1] = TR(lb[3], 28672);
    SB();
    MMA("", acc[2][2], bf[2], a[2][4], a[2][5]);
    SB();
    bh[4][1] = TR(lb[4], 28672);
    SB();
    LDA(a[0][3], ao0[0], 3072);
    SB();
    MMA("", acc[3][2], bf[2], a[2][6], a[2][7]);
    SB();
    bh[5][1] = TR(lb[5], 28672);
    SB();
    MMA("", acc[4][2], bf[2], a[2][8], a[2][9]);
    SB();
    bh[6][1] = TR(lb[6], 28672);
    SB();
    LDA(a[0][4], ao2[0], 0);
    SB();
    MMA("", acc[0][3], bf[3], a[2][0], a[2][1]);
    SB();
    bh[7][1] = TR(lb[7], 28672);
    SB();
    MMA("", acc[1][3], bf[3], a[2][2], a[2][3]);
    SB();
    bh[0][2] = TR(lb[0], 32768);
    SB();
    LDA(a[0][5], ao2[0], 1024);
    SB();
    MMA("", acc[2][3], bf[3], a[2][4], a[2][5]);
    SB();
    bh[1][2] = TR(lb[1], 32768);
    SB();
    MMA("", acc[3][3], bf[3], a[2][6], a[2][7]);
    SB();
    bh[2][2] = TR(lb[2], 32768);
    SB();
    LDA(a[0][6], ao2[0], 2048);
    SB();
    MMA("", acc[4][3], bf[3], a[2][8], a[2][9]);
    SB();
    bh[3][2] = TR(lb[3], 32768);
    SB();
    LDA(a[0][7], ao2[0], 3072);
    SB();
    MMA("", acc[0][4], bf[4], a[2][0], a[2][1]);
    SB();
    bh[4][2] = TR(lb[4], 32768);
    SB();
    MMA("", acc[1][4], bf[4], a[2][2], a[2][3]);
    SB();
    bh[5][2] = TR(lb[5], 32768);
    SB();
    LDA(a[0][8], ao4[0], 0);
    SB();
    MMA("", acc[2][4], bf[4], a[2][4], a[2][5]);
    SB();
    bh[6][2] = TR(lb[6], 32768);
    SB();
    MMA("", acc[3][4], bf[4], a[2][6], a[2][7]);
    SB();
    bh[7][2] = TR(lb[7], 32768);
    SB();
    LDA(a[0][9], ao4[0], 1024);
    SB();
    MMA("", acc[4][4], bf[4], a[2][8], a[2][9]);
    SB();
    asm volatile("s_waitcnt vmcnt(21)" ::: "memory");
    bh[0][3] = TR(lb[0], 36864);
    SB();
    MMA("", acc[0][5], bf[5], a[2][0], a[2][1]);
    SB();
    bh[1][3] = TR(lb[1], 36864);
    SB();
    DMAC(11264, voffW1, WK(4), 458752);
    SB();
    MMA("", acc[1][5], bf[5], a[2][2], a[2][3]);
    SB();
    bh[2][3] = TR(lb[2], 36864);
    SB();
    MMA("", acc[2][5], bf[5], a[2][4], a[2][5]);
    SB();
    bh[3][3] = TR(lb[3], 36864);
    SB();
    asm volatile("s_waitcnt lgkmcnt(15)" ::: "memory");
    DMAC(12288, voffW0, WK(4), 294912);
    SB();
    MMA("", acc[3][5], bf[5], a[2][6], a[2][7]);
    SB();
    bh[4][3] = TR(lb[4], 36864);
    SB();
    MMA("", acc[4][5], bf[5], a[2][8], a[2][9]);
    SB();
    bh[5][3] = TR(lb[5], 36864);
    SB();
    DMAC(13312, voffW1, WK(4), 360448);
    SB();
    MMA("", acc[0][6], bf[6], a[2][0], a[2][1]);
    SB();
    bh[6][3] = TR(lb[6], 36864);
    SB();
    MMA("", acc[1][6], bf[6], a[2][2], a[2][3]);
    SB();
    bh[7][3] = TR(lb[7], 36864);
    SB();
    DMAC(14336, voffW0, WK(4), 425984);
    SB();
    MMA("", acc[2][6], bf[6], a[2][4], a[2][5]);
    SB();
    asm volatile("s_waitcnt vmcnt(12)" : "+v"(a[0][0]), "+v"(a[0][1]) :: "memory");
    am = MINBLK(0, 0);
    asm volatile("" : "+v"(am));
    SB();
    MMA("", acc[3][6], bf[6], a[2][6], a[2][7]);
    SB();
    asm volatile("s_waitcnt vmcnt(10)" : "+v"(a[0][2]), "+v"(a[0][3]) :: "memory");
    am = min(am, MINBLK(0, 1));
    asm volatile("" : "+v"(am));
    SB();
    MMA("", acc[4][6], bf[6], a[2][8], a[2][9]);
    SB();
    asm volatile("s_waitcnt vmcnt(8)" : "+v"(a[0][4]), "+v"(a[0][5]) :: "memory");
    am = min(am, MINBLK(0, 2));
    asm volatile("" : "+v"(am));
    SB();
    MMA("", acc[0][7], bf[7], a[2][0], a[2][1]);
    SB();
    asm volatile("s_waitcnt vmcnt(6)" : "+v"(a[0][6]), "+v"(a[0][7]) :: "memory");
    am = min(am, MINBLK(0, 3));
    asm volatile("" : "+v"(am));
    SB();
    MMA("", acc[1][7], bf[7], a[2][2], a[2][3]);
    SB();
    asm volatile("s_waitcnt vmcnt(4)" : "+v"(a[0][8]), "+v"(a[0][9]) :: "memory");
    am = min(am, (MINBLK(0, 4) | no5));
    asm volatile("" : "+v"(am));
    SB();
    MMA("", acc[2][7], bf[7], a[2][4], a[2][5]);
    SB();
    SPIN(LDA(a[0][0], ao0[0], 0); LDA(a[0][1], ao0[0], 1024); LDA(a[0][2], ao0[0], 2048); LDA(a[0][3], ao0[0], 3072); LDA(a[0][4], ao2[0], 0); LDA(a[0][5], ao2[0], 1024); LDA(a[0][6], ao2[0], 2048); LDA(a[0][7], ao2[0], 3072); LDA(a[0][8], ao4[0], 0); LDA(a[0][9], ao4[0], 1024); asm volatile("s_waitcnt vmcnt(0)" : "+v"(a[0][0]), "+v"(a[0][1]), "+v"(a[0][2]), "+v"(a[0][3]), "+v"(a[0][4]), "+v"(a[0][5]), "+v"(a[0][6]), "+v"(a[0][7]), "+v"(a[0][8]), "+v"(a[0][9]) :: "memory"); am = min(min(min(min(MINBLK(0, 0), MINBLK(0, 1)), MINBLK(0, 2)), MINBLK(0, 3)), (MINBLK(0, 4) | no5)););
    SB();
    MMA("", acc[3][7], bf[7], a[2][6], a[2][7]);
    SB();
    DMAC(15360, voffW1, WK(4), 491520);
    SB();
    MMA("", acc[4][7], bf[7], a[2][8], a[2][9]);
    SB();
    asm volatile("s_waitcnt lgkmcnt(15)" ::: "memory");
    DMAC(16384, voffW0, WK(5), 0);
    SB();
    STAMP(5);
    MMA("", acc[0][0], bh[0], a[0][0], a[0][1]);
    SB();
    bf[0][0] = TR(lb[0], 0);
    SB();
    DMAC(17408, voffW1, WK(5), 65536);
    SB();
    MMA("", acc[1][0], bh[0], a[0][2], a[0][3]);
    SB();
    bf[1][0] = TR(lb[1], 0);
    SB();
    DMAC(18432, voffW0, WK(5), 131072);
    SB();
    MMA("", acc[2][0], bh[0], a[0][4], a[0][5]);
    SB();
    bf[2][0] = TR(lb[2], 0);
    SB();
    DMAC(19456, voffW1, WK(5), 196608);
    SB();
    MMA("", acc[3][0], bh[0], a[0][6], a[0][7]);
    SB();
    bf[3][0] = TR(lb[3], 0);
    SB();
    asm volatile("s_waitcnt lgkmcnt(15)" ::: "memory");
    DMAC(20480, voffW0, WK(5), 32768);
    SB();
    MMA("", acc[4][0], bh[0], a[0][8], a[0][9]);
    SB();
    bf[4][0] = TR(lb[4], 0);
    SB();
    DMAC(21504, voffW1, WK(5), 98304);
    SB();
    MMA("", acc[0][1], bh[1], a[0][0], a[0][1]);
    SB();
    bf[5][0] = TR(lb[5], 0);
    SB();
    DMAC(22528, voffW0, WK(5), 163840);
    SB();
    MMA("", acc[1][1], bh[1], a[0][2], a[0][3]);
    SB();
    bf[6][0] = TR(lb[6], 0);
    SB();
    MMA("", acc[2][1], bh[1], a[0][4], a[0][5]);
    SB();
    bf[7][0] = TR(lb[7], 0);
    SB();
    SETA(4);
    LDA(a[1][0], ao0[1], 0);
    SB();
    MMA("", acc[3][1], bh[1], a[0][6], a[0][7]);
    SB();
    bf[0][1] = TR(lb[0], 4096);
    SB();
    MMA("", acc[4][1], bh[1], a[0][8], a[0][9]);
    SB();
    bf[1][1] = TR(lb[1], 4096);
    SB();
    LDA(a[1][1], ao0[1], 1024);
    SB();
    MMA("", acc[0][2], bh[2], a[0][0], a[0][1]);
    SB();
    bf[2][1] = TR(lb[2], 4096);
    SB();
    MMA("", acc[1][2], bh[2], a[0][2], a[0][3]);
    SB();
    bf[3][1] = TR(lb[3], 4096);
    SB();
    LDA(a[1][2], ao0[1], 2048);
    SB();
    MMA("", acc[2][2], bh[2], a[0][4], a[0][5]);
    SB();
    bf[4][1] = TR(lb[4], 4096);
    SB();
    MMA("", acc[3][2], bh[2], a[0][6], a[0][7]);
    SB();
    bf[5][1] = TR(lb[5], 4096);
    SB();
    LDA(a[1][3], ao0[1], 3072);
    SB();
    MMA("", acc[4][2], bh[2], a[0][8], a[0][9]);
    SB();
    bf[6][1] = TR(lb[6], 4096);
    SB();
    LDA(a[1][4], ao2[1], 0);
    SB();
    MMA("", acc[0][3], bh[3], a[0][0], a[0][1]);
    SB();
    bf[7][1] = TR(lb[7], 4096);
    SB();
    MMA("", acc[1][3], bh[3], a[0][2], a[0][3]);
    SB();
    asm volatile("s_waitcnt vmcnt(16)" ::: "memory");
    bf[0][2] = TR(lb[0], 8192);
    SB();
    LDA(a[1][5], ao2[1], 1024);
    SB();
    MMA("", acc[2][3], bh[3], a[0][4], a[0][5]);
    SB();
    bf[1][2] = TR(lb[1], 8192);
    SB();
    MMA("", acc[3][3], bh[3], a[0][6], a[0][7]);
    SB();
    bf[2][2] = TR(lb[2], 8192);
    SB();
    LDA(a[1][6], ao2[1], 2048);
    SB();
    MMA("", acc[4][3], bh[3], a[0][8], a[0][9]);
    SB();
    bf[3][2] = TR(lb[3], 8192);
    SB();
    MMA("", acc[0][4], bh[4], a[0][0], a[0][1]);
    SB();
    bf[4][2] = TR(lb[4], 8192);
    SB();
    LDA(a[1][7], ao2[1], 3072);
    SB();
    MMA("", acc[1][4], bh[4], a[0][2], a[0][3]);
    SB();
    bf[5][2] = TR(lb[5], 8192);
    SB();
    MMA("", acc[2][4], bh[4], a[0][4], a[0][5]);
    SB();
    bf[6][2] = TR(lb[6], 8192);
    SB();
    LDA(a[1][8], ao4[1], 0);
    SB();
    MMA("", acc[3][4], bh[4], a[0][6], a[0][7]);
    SB();
    bf[7][2] = TR(lb[7], 8192);
    SB();
    LDA(a[1][9], ao4[1], 1024);
    SB();
    MMA("", acc[4][4], bh[4], a[0][8], a[0][9]);
    SB();
    asm volatile("s_waitcnt vmcnt(17)" ::: "memory");
    bf[0][3] = TR(lb[0], 12288);
    SB();
    MMA("", acc[0][5], bh[5], a[0][0], a[0][1]);
    SB();
    bf[1][3] = TR(lb[1], 12288);
    SB();
    DMAC(23552, voffW1, WK(5), 229376);
    SB();
    MMA("", acc[1][5], bh[5], a[0][2], a[0][3]);
    SB();
    bf[2][3] = TR(lb[2], 12288);
    SB();
    MMA("", acc[2][5], bh[5], a[0][4], a[0][5]);
    SB();
    bf[3][3] = TR(lb[3], 12288);
    SB();
    asm volatile("s_waitcnt lgkmcnt(15)" ::: "memory");
    DMAC(24576, voffW0, WK(5), 262144);
    SB();
    MMA("", acc[3][5], bh[5], a[0][6], a[0][7]);
    SB();
    bf[4][3] = TR(lb[4], 12288);
    SB();
    MMA("", acc[4][5], bh[5], a[0][8], a[0][9]);
    SB();
    bf[5][3] = TR(lb[5], 12288);
    SB();
    DMAC(25600, voffW1, WK(5), 327680);
    SB();
    MMA("", acc[0][6], bh[6], a[0][0], a[0][1]);
    SB();
    bf[6][3] = TR(lb[6], 12288);
    SB();
    MMA("", acc[1][6], bh[6], a[0][2], a[0][3]);
    SB();
    bf[7][3] = TR(lb[7], 12288);
    SB();
    DMAC(26624, voffW0, WK(5), 393216);
    SB();
    MMA("", acc[2][6], bh[6], a[0][4], a[0][5]);
    SB();
    asm volatile("s_waitcnt vmcnt(12)" : "+v"(a[1][0]), "+v"(a[1][1]) :: "memory");
    am = MINBLK(1, 0);
    asm volatile("" : "+v"(am));
    SB();
    MMA("", acc[3][6], bh[6], a[0][6], a[0][7]);
    SB();
    asm volatile("s_waitcnt vmcnt(10)" : "+v"(a[1][2]), "+v"(a[1][3]) :: "memory");
    am = min(am, MINBLK(1, 1));
    asm volatile("" : "+v"(am));
    SB();
    MMA("", acc[4][6], bh[6], a[0][8], a[0][9]);
    SB();
    asm volatile("s_waitcnt vmcnt(8)" : "+v"(a[1][4]), "+v"(a[1][5]) :: "memory");
    am = min(am, MINBLK(1, 2));
    asm volatile("" : "+v"(am));
    SB();
    MMA("", acc[0][7], bh[7], a[0][0], a[0][1]);
    SB();
    asm volatile("s_waitcnt vmcnt(6)" : "+v"(a[1][6]), "+v"(a[1][7]) :: "memory");
    am = min(am, MINBLK(1, 3));
    asm volatile("" : "+v"(am));
    SB();
    MMA("", acc[1][7], bh[7], a[0][2], a[0][3]);
    SB();
    asm volatile("s_waitcnt vmcnt(4)" : "+v"(a[1][8]), "+v"(a[1][9]) :: "memory");
    am = min(am, (MINBLK(1, 4) | no5));
    asm volatile("" : "+v"(am));
    SB();
    MMA("", acc[2][7], bh[7], a[0][4], a[0][5]);
    SB();
    SPIN(LDA(a[1][0], ao0[1], 0); LDA(a[1][1], ao0[1], 1024); LDA(a[1][2], ao0[1], 2048); LDA(a[1][3], ao0[1], 3072); LDA(a[1][4], ao2[1], 0); LDA(a[1][5], ao2[1], 1024); LDA(a[1][6], ao2[1], 2048); LDA(a[1][7], ao2[1], 3072); LDA(a[1][8], ao4[1], 0); LDA(a[1][9], ao4[1], 1024); asm volatile("s_waitcnt vmcnt(0)" : "+v"(a[1][0]), "+v"(a[1][1]), "+v"(a[1][2]), "+v"(a[1][3]), "+v"(a[1][4]), "+v"(a[1][5]), "+v"(a[1][6]), "+v"(a[1][7]), "+v"(a[1][8]), "+v"(a[1][9]) :: "memory"); am = min(min(min(min(MINBLK(1, 0), MINBLK(1, 1)), MINBLK(1, 2)), MINBLK(1, 3)), (MINBLK(1, 4) | no5)););
    SB();
    MMA("", acc[3][7], bh[7], a[0][6], a[0][7]);
    SB();
    DMAC(27648, voffW1, WK(5), 458752);
    SB();
    MMA("", acc[4][7], bh[7], a[0][8], a[0][9]);
    SB();
    asm volatile("s_waitcnt lgkmcnt(15)" ::: "memory");
    DMAC(28672, voffW0, WK(5), 294912);
    SB();
    STAMP(6);
    MMA("", acc[0][0], bf[0], a[1][0], a[1][1]);
    SB();
    bh[0][0] = TR(lb[0], 16384);
    SB();
    DMAC(29696, voffW1, WK(5), 360448);
    SB();
    MMA("", acc[1][0], bf[0], a[1][2], a[1][3]);
    SB();
    bh[1][0] = TR(lb[1], 16384);
    SB();
    DMAC(30720, voffW0, WK(5), 425984);
    SB();
    MMA("", acc[2][0], bf[0], a[1][4], a[1][5]);
    SB();
    bh[2][0] = TR(lb[2], 16384);
    SB();
    DMAC(31744, voffW1, WK(5), 491520);
    SB();
    MMA("", acc[3][0], bf[0], a[1][6], a[1][7]);
    SB();
    bh[3][0] = TR(lb[3], 16384);
    SB();
    asm volatile("s_waitcnt lgkmcnt(15)" ::: "memory");
    DMAC(32768, voffW0, WK(6), 0);
    SB();
    MMA("", acc[4][0], bf[0], a[1][8], a[1][9]);
    SB();
    bh[4][0] = TR(lb[4], 16384);
    SB();
    DMAC(33792, voffW1, WK(6), 65536);
    SB();
    MMA("", acc[0][1], bf[1], a[1][0], a[1][1]);
    SB();
    bh[5][0] = TR(lb[5], 16384);
    SB();
    DMAC(34816, voffW0, WK(6), 131072);
    SB();
    MMA("", acc[1][1], bf[1], a[1][2], a[1][3]);
    SB();
    bh[6][0] = TR(lb[6], 16384);
    SB();
    MMA("", acc[2][1], bf[1], a[1][4], a[1][5]);
    SB();
    bh[7][0] = TR(lb[7], 16384);
    SB();
    SETA(5);
    LDA(a[2][0], ao0[2], 0);
    SB();
    MMA("", acc[3][1], bf[1], a[1][6], a[1][7]);
    SB();
    asm volatile("s_waitcnt vmcnt(12)" ::: "memory");
    bh[0][1] = TR(lb[0], 20480);
    SB();
    MMA("", acc[4][1], bf[1], a[1][8], a[1][9]);
    SB();
    bh[1][1] = TR(lb[1], 20480);
    SB();
    LDA(a[2][1], ao0[2], 1024);
    SB();
    MMA("", acc[0][2], bf[2], a[1][0], a[1][1]);
    SB();
    bh[2][1] = TR(lb[2], 20480);
    SB();
    MMA("", acc[1][2], bf[2], a[1][2], a[1][3]);
    SB();
    bh[3][1] = TR(lb[3], 20480);
    SB();
    LDA(a[2][2], ao0[2], 2048);
    SB();
    MMA("", acc[2][2], bf[2], a[1][4], a[1][5]);
    SB();
    bh[4][1] = TR(lb[4], 20480);
    SB();
    MMA("", acc[3][2], bf[2], a[1][6], a[1][7]);
    SB();
    bh[5][1] = TR(lb[5], 20480);
    SB();
    LDA(a[2][3], ao0[2], 3072);
    SB();
    MMA("", acc[4][2], bf[2], a[1][8], a[1][9]);
    SB();
    bh[6][1] = TR(lb[6], 20480);
    SB();
    MMA("", acc[0][3], bf[3], a[1][0], a[1][1]);
    SB();
    bh[7][1] = TR(lb[7], 20480);
    SB();
    LDA(a[2][4], ao2[2], 0);
    SB();
    MMA("", acc[1][3], bf[3], a[1][2], a[1][3]);
    SB();
    asm volatile("s_waitcnt vmcnt(12)" ::: "memory");
    bh[0][2] = TR(lb[0], 24576);
    SB();
    LDA(a[2][5], ao2[2], 1024);
    SB();
    MMA("", acc[2][3], bf[3], a[1][4], a[1][5]);
    SB();
    bh[1][2] = TR(lb[1], 24576);
    SB();
    MMA("", acc[3][3], bf[3], a[1][6], a[1][7]);
    SB();
    bh[2][2] = TR(lb[2], 24576);
    SB();
    LDA(a[2][6], ao2[2], 2048);
    SB();
    MMA("", acc[4][3], bf[3], a[1][8], a[1][9]);
    SB();
    bh[3][2] = TR(lb[3], 24576);
    SB();
    MMA("", acc[0][4], bf[4], a[1][0], a[1][1]);
    SB();
    bh[4][2] = TR(lb[4], 24576);
    SB();
    LDA(a[2][7], ao2[2], 3072);
    SB();
    MMA("", acc[1][4], bf[4], a[1][2], a[1][3]);
    SB();
    bh[5][2] = TR(lb[5], 24576);
    SB();
    MMA("", acc[2][4], bf[4], a[1][4], a[1][5]);
    SB();
    bh[6][2] = TR(lb[6], 24576);
    SB();
    LDA(a[2][8], ao4[2], 0);
    SB();
    MMA("", acc[3][4], bf[4], a[1][6], a[1][7]);
    SB();
    bh[7][2] = TR(lb[7], 24576);
    SB();
    MMA("", acc[4][4], bf[4], a[1][8], a[1][9]);
    SB();
    asm volatile("s_waitcnt vmcnt(12)" ::: "memory");
    bh[0][3] = TR(lb[0], 28672);
    SB();
    LDA(a[2][9], ao4[2], 1024);
    SB();
    MMA("", acc[0][5], bf[5], a[1][0], a[1][1]);
    SB();
    bh[1][3] = TR(lb[1], 28672);
    SB();
    DMAC(35840, voffW1, WK(6), 196608);
    SB();
    MMA("", acc[1][5], bf[5], a[1][2], a[1][3]);
    SB();
    bh[2][3] = TR(lb[2], 28672);
    SB();
    MMA("", acc[2][5], bf[5], a[1][4], a[1][5]);
    SB();
    bh[3][3] = TR(lb[3], 28672);
    SB();
    asm volatile("s_waitcnt lgkmcnt(15)" ::: "memory");
    DMAC(36864, voffW0, WK(6), 32768);
    SB();
    MMA("", acc[3][5], bf[5], a[1][6], a[1][7]);
    SB();
    bh[4][3] = TR(lb[4], 28672);
    SB();
    MMA("", acc[4][5], bf[5], a[1][8], a[1][9]);
    SB();
    bh[5][3] = TR(lb[5], 28672);
    SB();
    DMAC(37888, voffW1, WK(6), 98304);
    SB();
    MMA("", acc[0][6], bf[6], a[1][0], a[1][1]);
    SB();
    bh[6][3] = TR(lb[6], 28672);
    SB();
    MMA("", acc[1][6], bf[6], a[1][2], a[1][3]);
    SB();
    bh[7][3] = TR(lb[7], 28672);
    SB();
    DMAC(38912, voffW0, WK(6), 163840);
    SB();
    MMA("", acc[2][6], bf[6], a[1][4], a[1][5]);
    SB();
    asm volatile("s_waitcnt vmcnt(12)" : "+v"(a[2][0]), "+v"(a[2][1]) :: "memory");
    am = MINBLK(2, 0);
    asm volatile("" : "+v"(am));
    SB();
    MMA("", acc[3][6], bf[6], a[1][6], a[1][7]);
    SB();
    asm volatile("s_waitcnt vmcnt(10)" : "+v"(a[2][2]), "+v"(a[2][3]) :: "memory");
    am = min(am, MINBLK(2, 1));
    asm volatile("" : "+v"(am));
    SB();
    MMA("", acc[4][6], bf[6], a[1][8], a[1][9]);
    SB();
    asm volatile("s_waitcnt vmcnt(8)" : "+v"(a[2][4]), "+v"(a[2][5]) :: "memory");
    am = min(am, MINBLK(2, 2));
    asm volatile("" : "+v"(am));
    SB();
    MMA("", acc[0][7], bf[7], a[1][0], a[1][1]);
    SB();
    asm volatile("s_waitcnt vmcnt(6)" : "+v"(a[2][6]), "+v"(a[2][7]) :: "memory");
    am = min(am, MINBLK(2, 3));
    asm volatile("" : "+v"(am));
    SB();
    MMA("", acc[1][7], bf[7], a[1][2], a[1][3]);
    SB();
    asm volatile("s_waitcnt vmcnt(4)" : "+v"(a[2][8]), "+v"(a[2][9]) :: "memory");
    am = min(am, (MINBLK(2, 4) | no5));
    asm volatile("" : "+v"(am));
    SB();
    DMAC(39936, voffW1, WK(6), 229376);
    SB();
    MMA("", acc[2][7], bf[7], a[1][4], a[1][5]);
    SB();
    SPIN(LDA(a[2][0], ao0[2], 0); LDA(a[2][1], ao0[2], 1024); LDA(a[2][2], ao0[2], 2048); LDA(a[2][3], ao0[2], 3072); LDA(a[2][4], ao2[2], 0); LDA(a[2][5], ao2[2], 1024); LDA(a[2][6], ao2[2], 2048); LDA(a[2][7], ao2[2], 3072); LDA(a[2][8], ao4[2], 0); LDA(a[2][9], ao4[2], 1024); asm volatile("s_waitcnt vmcnt(0)" : "+v"(a[2][0]), "+v"(a[2][1]), "+v"(a[2][2]), "+v"(a[2][3]), "+v"(a[2][4]), "+v"(a[2][5]), "+v"(a[2][6]), "+v"(a[2][7]), "+v"(a[2][8]), "+v"(a[2][9]) :: "memory"); am = min(min(min(min(MINBLK(2, 0), MINBLK(2, 1)), MINBLK(2, 2)), MINBLK(2, 3)), (MINBLK(2, 4) | no5)););
    SB();
    MMA("", acc[3][7], bf[7], a[1][6], a[1][7]);
    SB();
    asm volatile("s_waitcnt lgkmcnt(15)" ::: "memory");
    DMAC(0, voffW0, WK(6), 262144);
    SB();
    MMA("", acc[4][7], bf[7], a[1][8], a[1][9]);
    SB();
    DMAC(1024, voffW1, WK(6), 327680);
    SB();
    STAMP(7);
    MMA("", acc[0][0], bh[0], a[2][0], a[2][1]);
    SB();
    asm volatile("s_waitcnt vmcnt(6)" ::: "memory");
    bf[0][0] = TR(lb[0], 32768);
    SB();
    DMAC(2048, voffW0, WK(6), 393216);
    SB();
    MMA("", acc[1][0], bh[0], a[2][2], a[2][3]);
    SB();
    bf[1][0] = TR(lb[1], 32768);
    SB();
    DMAC(3072, voffW1, WK(6), 458752);
    SB();
    MMA("", acc[2][0], bh[0], a[2][4], a[2][5]);
    SB();
    bf[2][0] = TR(lb[2], 32768);
    SB();
    MMA("", acc[3][0], bh[0], a[2][6], a[2][7]);
    SB();
    bf[3][0] = TR(lb[3], 32768);
    SB();
    asm volatile("s_waitcnt lgkmcnt(15)" ::: "memory");
    DMAC(4096, voffW0, WK(6), 294912);
    SB();
    MMA("", acc[4][0], bh[0], a[2][8], a[2][9]);
    SB();
    bf[4][0] = TR(lb[4], 32768);
    SB();
    MMA("", acc[0][1], bh[1], a[2][0], a[2][1]);
    SB();
    bf[5][0] = TR(lb[5], 32768);
    BGV(0, 0);
    SB();
    MMA("", acc[1][1], bh[1], a[2][2], a[2][3]);
    SB();
    bf[6][0] = TR(lb[6], 32768);
    BGV(0, 1);
    SB();
    MMA("", acc[2][1], bh[1], a[2][4], a[2][5]);
    SB();
    bf[7][0] = TR(lb[7], 32768);
    SB();
    DMAC(5120, voffW1, WK(6), 360448);
    SB();
    MMA("", acc[3][1], bh[1], a[2][6], a[2][7]);
    SB();
    asm volatile("s_waitcnt vmcnt(6)" ::: "memory");
    bf[0][1] = TR(lb[0], 36864);
    SB();
    DMAC(6144, voffW0, WK(6), 425984);
    SB();
    MMA("", acc[4][1], bh[1], a[2][8], a[2][9]);
    SB();
    bf[1][1] = TR(lb[1], 36864);
    SB();
    MMA("", acc[0][2], bh[2], a[2][0], a[2][1]);
    SB();
    bf[2][1] = TR(lb[2], 36864);
    BGV(1, 0);
    SB();
    MMA("", acc[1][2], bh[2], a[2][2], a[2][3]);
    SB();
    bf[3][1] = TR(lb[3], 36864);
    BGV(1, 1);
    SB();
    MMA("", acc[2][2], bh[2], a[2][4], a[2][5]);
    SB();
    bf[4][1] = TR(lb[4], 36864);
    SB();
    DMAC(7168, voffW1, WK(6), 491520);
    SB();
    MMA("", acc[3][2], bh[2], a[2][6], a[2][7]);
    SB();
    bf[5][1] = TR(lb[5], 36864);
    SB();
    SETA(6);
    LDA(a[0][0], ao0[0], 0);
    SB();
    MMA("", acc[4][2], bh[2], a[2][8], a[2][9]);
    SB();
    bf[6][1] = TR(lb[6], 36864);
    SB();
    LDA(a[0][1], ao0[0], 1024);
    SB();
    MMA("", acc[0][3], bh[3], a[2][0], a[2][1]);
    SB();
    bf[7][1] = TR(lb[7], 36864);
    BGV(2, 0);
    SB();
    MMA("", acc[1][3], bh[3], a[2][2], a[2][3]);
    SB();
    asm volatile("s_waitcnt vmcnt(6)" ::: "memory");
    bf[0][2] = TR(lb[0], 0);
    BGV(2, 1);
    SB();
    MMA("", acc[2][3], bh[3], a[2][4], a[2][5]);
    SB();
    bf[1][2] = TR(lb[1], 0);
    SB();
    SETA(7);
    LDA(a[1][0], ao0[1], 0);
    SB();
    MMA("", acc[3][3], bh[3], a[2][6], a[2][7]);
    SB();
    bf[2][2] = TR(lb[2], 0);
    SB();
    MMA("", acc[4][3], bh[3], a[2][8], a[2][9]);
    SB();
    bf[3][2] = TR(lb[3], 0);
    SB();
    LDA(a[1][1], ao0[1], 1024);
    SB();
    MMA("", acc[0][4], bh[4], a[2][0], a[2][1]);
    SB();
    bf[4][2] = TR(lb[4], 0);
    BGV(3, 0);
    SB();
    MMA("", acc[1][4], bh[4], a[2][2], a[2][3]);
    SB();
    bf[5][2] = TR(lb[5], 0);
    BGV(3, 1);
    SB();
    MMA("", acc[2][4], bh[4], a[2][4], a[2][5]);
    SB();
    bf[6][2] = TR(lb[6], 0);
    SB();
    LDA(a[0][2], ao0[0], 2048);
    SB();
    MMA("", acc[3][4], bh[4], a[2][6], a[2][7]);
    SB();
    bf[7][2] = TR(lb[7], 0);
    SB();
    MMA("", acc[4][4], bh[4], a[2][8], a[2][9]);
    SB();
    asm volatile("s_waitcnt vmcnt(5)" ::: "memory");
    bf[0][3] = TR(lb[0], 4096);
    SB();
    MMA("", acc[0][5], bh[5], a[2][0], a[2][1]);
    SB();
    bf[1][3] = TR(lb[1], 4096);
    BGV(4, 0);
    SB();
    MMA("", acc[1][5], bh[5], a[2][2], a[2][3]);
    SB();
    bf[2][3] = TR(lb[2], 4096);
    BGV(4, 1);
    SB();
    MMA("", acc[2][5], bh[5], a[2][4], a[2][5]);
    SB();
    bf[3][3] = TR(lb[3], 4096);
    SB();
    LDA(a[0][3], ao0[0], 3072);
    SB();
    MMA("", acc[3][5], bh[5], a[2][6], a[2][7]);
    SB();
    bf[4][3] = TR(lb[4], 4096);
    SB();
    LDA(a[1][2], ao0[1], 2048);
    SB();
    MMA("", acc[4][5], bh[5], a[2][8], a[2][9]);
    SB();
    bf[5][3] = TR(lb[5], 4096);
    SB();
    MMA("", acc[0][6], bh[6], a[2][0], a[2][1]);
    SB();
    bf[6][3] = TR(lb[6], 4096);
    BGV(5, 0);
    SB();
    MMA("", acc[1][6], bh[6], a[2][2], a[2][3]);
    SB();
    bf[7][3] = TR(lb[7], 4096);
    BGV(5, 1);
    SB();
    MMA("", acc[2][6], bh[6], a[2][4], a[2][5]);
    SB();
    LDA(a[1][3], ao0[1], 3072);
    SB();
    MMA("", acc[3][6], bh[6], a[2][6], a[2][7]);
    SB();
    MMA("", acc[4][6], bh[6], a[2][8], a[2][9]);
    SB();
    MMA("", acc[0][7], bh[7], a[2][0], a[2][1]);
    SB();
    asm volatile("s_waitcnt vmcnt(6)" : "+v"(a[0][0]), "+v"(a[0][1]) :: "memory");
    am = MINBLK(0, 0);
    asm volatile("" : "+v"(am));
    SB();
    BGV(6, 0);
    SB();
    MMA("", acc[1][7], bh[7], a[2][2], a[2][3]);
    SB();
    BGV(6, 1);
    SB();
    MMA("", acc[2][7], bh[7], a[2][4], a[2][5]);
    SB();
    asm volatile("s_waitcnt vmcnt(4)" : "+v"(a[1][0]), "+v"(a[1][1]) :: "memory");
    am = min(am, MINBLK(1, 0));
    asm volatile("" : "+v"(am));
    SB();
    MMA("", acc[3][7], bh[7], a[2][6], a[2][7]);
    SB();
    SPIN(LDA(a[0][0], ao0[0], 0); LDA(a[0][1], ao0[0], 1024); LDA(a[0][2], ao0[0], 2048); LDA(a[0][3], ao0[0], 3072); LDA(a[1][0], ao0[1], 0); LDA(a[1][1], ao0[1], 1024); LDA(a[1][2], ao0[1], 2048); LDA(a[1][3], ao0[1], 3072); asm volatile("s_waitcnt vmcnt(0)" : "+v"(a[0][0]), "+v"(a[0][1]), "+v"(a[0][2]), "+v"(a[0][3]), "+v"(a[1][0]), "+v"(a[1][1]), "+v"(a[1][2]), "+v"(a[1][3]) :: "memory"); am = min(MINBLK(0, 0), MINBLK(1, 0)););
    SB();
    MMA("", acc[4][7], bh[7], a[2][8], a[2][9]);
    SB();
    LDA(a[0][4], ao2[0], 0);
    SB();
    STAMP(8);
    MMA(NOP, acc[0][0], bf[0], a[0][0], a[0][1]);
    SB();
    BGV(7, 0);
    SB();
    MMA(NOP, acc[0][0], bg[0], a[1][0], a[1][1]);
    SB();
    BGV(7, 1);
    SB();
    MMA("", acc[0][1], bf[1], a[0][0], a[0][1]);
    SB();
    LDA(a[0][5], ao2[0], 1024);
    SB();
    MMA("", acc[0][1], bg[1], a[1][0], a[1][1]);
    SB();
    PUT(0, 0);
    SB();
    MMA("", acc[0][2], bf[2], a[0][0], a[0][1]);
    SB();
    LDA(a[1][4], ao2[1], 0);
    SB();
    MMA("", acc[0][2], bg[2], a[1][0], a[1][1]);
    SB();
    PUT(0, 1);
    SB();
    MMA("", acc[0][3], bf[3], a[0][0], a[0][1]);
    SB();
    LDA(a[1][5], ao2[1], 1024);
    SB();
    MMA("", acc[0][3], bg[3], a[1][0], a[1][1]);
    SB();
    PUT(0, 2);
    SB();
    MMA("", acc[0][4], bf[4], a[0][0], a[0][1]);
    SB();
    MMA("", acc[0][4], bg[4], a[1][0], a[1][1]);
    SB();
    PUT(0, 3);
    SB();
    MMA("", acc[0][5], bf[5], a[0][0], a[0][1]);
    SB();
    MMA("", acc[0][5], bg[5], a[1][0], a[1][1]);
    SB();
    asm volatile("s_waitcnt vmcnt(6)" : "+v"(a[0][2]), "+v"(a[0][3]) :: "memory");
    am = MINBLK(0, 1);
    asm volatile("" : "+v"(am));
    SB();
    PUT(0, 4);
    SB();
    MMA("", acc[0][6], bf[6], a[0][0], a[0][1]);
    SB();
    MMA("", acc[0][6], bg[6], a[1][0], a[1][1]);
    SB();
    asm volatile("s_waitcnt vmcnt(4)" : "+v"(a[1][2]), "+v"(a[1][3]) :: "memory");
    am = min(am, MINBLK(1, 1));
    asm volatile("" : "+v"(am));
    SB();
    PUT(0, 5);
    SB();
    MMA("", acc[0][7], bf[7], a[0][0], a[0][1]);
    SB();
    SPIN(LDA(a[0][2], ao0[0], 2048); LDA(a[0][3], ao0[0], 3072); LDA(a[0][4], ao2[0], 0); LDA(a[0][5], ao2[0], 1024); LDA(a[1][2], ao0[1], 2048); LDA(a[1][3], ao0[1], 3072); LDA(a[1][4], ao2[1], 0); LDA(a[1][5], ao2[1], 1024); asm volatile("s_waitcnt vmcnt(0)" : "+v"(a[0][2]), "+v"(a[0][3]), "+v"(a[0][4]), "+v"(a[0][5]), "+v"(a[1][2]), "+v"(a[1][3]), "+v"(a[1][4]), "+v"(a[1][5]) :: "memory"); am = min(MINBLK(0, 1), MINBLK(1, 1)););
    SB();
    MMA("", acc[0][7], bg[7], a[1][0], a[1][1]);
    SB();
    PUT(0, 6);
    SB();
    MMA(NOP, acc[1][0], bf[0], a[0][2], a[0][3]);
    SB();
    LDA(a[0][6], ao2[0], 2048);
    SB();
    MMA(NOP, acc[1][0], bg[0], a[1][2], a[1][3]);
    SB();
    PUT(0, 7);
    SB();
    MMA("", acc[1][1], bf[1], a[0][2], a[0][3]);
    SB();
    LDA(a[0][7], ao2[0], 3072);
    SB();
    MMA("", acc[1][1], bg[1], a[1][2], a[1][3]);
    SB();
    asm volatile("s_waitcnt lgkmcnt(8)" ::: "memory");
    PUT(1, 0);
    SB();
    MMA("", acc[1][2], bf[2], a[0][2], a[0][3]);
    SB();
    BAR(1);
    SB();
    GET(0, 0, 0);
    GET(0, 0, 1);
    SB();
    GET(0, 0, 2);
    GET(0, 0, 3);
    SB();
    MMA("", acc[1][2], bg[2], a[1][2], a[1][3]);
    SB();
    PUT(1, 1);
    SB();
    MMA("", acc[1][3], bf[3], a[0][2], a[0][3]);
    SB();
    GET(0, 1, 0);
    GET(0, 1, 1);
    SB();
    GET(0, 1, 2);
    GET(0, 1, 3);
    SB();
    MMA("", acc[1][3], bg[3], a[1][2], a[1][3]);
    SB();
    PUT(1, 2);
    SB();
    MMA("", acc[1][4], bf[4], a[0][2], a[0][3]);
    SB();
    asm volatile("s_waitcnt lgkmcnt(8)" : "+v"(gv[0][0]), "+v"(gv[0][1]) :: "memory");
    ADD4(0, 0, 1);
    SB();
    asm volatile("s_waitcnt lgkmcnt(6)" : "+v"(gv[0][2]), "+v"(gv[0][3]) :: "memory");
    ADD4(0, 2, 3);
    SB();
    MMA("", acc[1][4], bg[4], a[1][2], a[1][3]);
    SB();
    ADD4(0, 0, 2);
    SB();
    PUT(1, 3);
    SB();
    LDA(a[0][8], ao4[0], 0);
    SB();
    MMA("", acc[1][5], bf[5], a[0][2], a[0][3]);
    SB();
    MUL4(0);
    SB();
    OUT(0, 0);
    SB();
    LDA(a[0][9], ao4[0], 1024);
    SB();
    MMA("", acc[1][5], bg[5], a[1][2], a[1][3]);
    SB();
    asm volatile("s_waitcnt vmcnt(7)" : "+v"(a[0][4]), "+v"(a[0][5]) :: "memory");
    am = MINBLK(0, 2);
    asm volatile("" : "+v"(am));
    SB();
    PUT(1, 4);
    SB();
    LDA(a[1][6], ao2[1], 2048);
    SB();
    MMA("", acc[1][6], bf[6], a[0][2], a[0][3]);
    SB();
    asm volatile("s_waitcnt lgkmcnt(5)" : "+v"(gv[1][0]), "+v"(gv[1][1]) :: "memory");
    ADD4(1, 0, 1);
    SB();
    asm volatile("s_waitcnt lgkmcnt(3)" : "+v"(gv[1][2]), "+v"(gv[1][3]) :: "memory");
    ADD4(1, 2, 3);
    SB();
    LDA(a[1][7], ao2[1], 3072);
    SB();
    MMA("", acc[1][6], bg[6], a[1][2], a[1][3]);
    SB();
    asm volatile("s_waitcnt vmcnt(7)" : "+v"(a[1][4]), "+v"(a[1][5]) :: "memory");
    am = min(am, MINBLK(1, 2));
    asm volatile("" : "+v"(am));
    SB();
    ADD4(1, 0, 2);
    SB();
    PUT(1, 5);
    SB();
    LDA(a[1][8], ao4[1], 0);
    SB();
    MMA("", acc[1][7], bf[7], a[0][2], a[0][3]);
    SB();
    SPIN(LDA(a[0][4], ao2[0], 0); LDA(a[0][5], ao2[0], 1024); LDA(a[0][6], ao2[0], 2048); LDA(a[0][7], ao2[0], 3072); LDA(a[0][8], ao4[0], 0); LDA(a[0][9], ao4[0], 1024); LDA(a[1][4], ao2[1], 0); LDA(a[1][5], ao2[1], 1024); LDA(a[1][6], ao2[1], 2048); LDA(a[1][7], ao2[1], 3072); LDA(a[1][8], ao4[1], 0); asm volatile("s_waitcnt vmcnt(0)" : "+v"(a[0][4]), "+v"(a[0][5]), "+v"(a[0][6]), "+v"(a[0][7]), "+v"(a[0][8]), "+v"(a[0][9]), "+v"(a[1][4]), "+v"(a[1][5]), "+v"(a[1][6]), "+v"(a[1][7]), "+v"(a[1][8]) :: "memory"); am = min(MINBLK(0, 2), MINBLK(1, 2)););
    SB();
    MUL4(1);
    SB();
    OUT(0, 1);
    SB();
    LDA(a[1][9], ao4[1], 1024);
    STSEQ();
    SB();
    MMA("", acc[1][7], bg[7], a[1][2], a[1][3]);
    SB();
    PUT(1, 6);
    SB();
    MMA(NOP, acc[2][0], bf[0], a[0][4], a[0][5]);
    SB();
    MMA(NOP, acc[2][0], bg[0], a[1][4], a[1][5]);
    SB();
    PUT(1, 7);
    SB();
    MMA("", acc[2][1], bf[1], a[0][4], a[0][5]);
    SB();
    MMA("", acc[2][1], bg[1], a[1][4], a[1][5]);
    SB();
    PUT(2, 0);
    SB();
    MMA("", acc[2][2], bf[2], a[0][4], a[0][5]);
    SB();
    BAR(1);
    SB();
    GET(1, 0, 0);
    GET(1, 0, 1);
    SB();
    GET(1, 0, 2);
    GET(1, 0, 3);
    SB();
    MMA("", acc[2][2], bg[2], a[1][4], a[1][5]);
    SB();
    PUT(2, 1);
    SB();
    MMA("", acc[2][3], bf[3], a[0][4], a[0][5]);
    SB();
    GET(1, 1, 0);
    GET(1, 1, 1);
    SB();
    GET(1, 1, 2);
    GET(1, 1, 3);
    SB();
    MMA("", acc[2][3], bg[3], a[1][4], a[1][5]);
    SB();
    PUT(2, 2);
    SB();
    STAMP(9);
    MMA("", acc[2][4], bf[4], a[0][4], a[0][5]);
    SB();
    asm volatile("s_waitcnt lgkmcnt(8)" : "+v"(gv[0][0]), "+v"(gv[0][1]) :: "memory");
    ADD4(0, 0, 1);
    SB();
    asm volatile("s_waitcnt lgkmcnt(6)" : "+v"(gv[0][2]), "+v"(gv[0][3]) :: "memory");
    ADD4(0, 2, 3);
    SB();
    MMA("", acc[2][4], bg[4], a[1][4], a[1][5]);
    SB();
    ADD4(0, 0, 2);
    SB();
    PUT(2, 3);
    SB();
    MMA("", acc[2][5], bf[5], a[0][4], a[0][5]);
    SB();
    MUL4(0);
    SB();
    OUT(1, 0);
    SB();
    MMA("", acc[2][5], bg[5], a[1][4], a[1][5]);
    SB();
    asm volatile("s_waitcnt vmcnt(10)" : "+v"(a[0][6]), "+v"(a[0][7]) :: "memory");
    am = MINBLK(0, 3);
    asm volatile("" : "+v"(am));
    SB();
    PUT(2, 4);
    SB();
    MMA("", acc[2][6], bf[6], a[0][4], a[0][5]);
    SB();
    asm volatile("s_waitcnt lgkmcnt(5)" : "+v"(gv[1][0]), "+v"(gv[1][1]) :: "memory");
    ADD4(1, 0, 1);
    SB();
    asm volatile("s_waitcnt lgkmcnt(3)" : "+v"(gv[1][2]), "+v"(gv[1][3]) :: "memory");
    ADD4(1, 2, 3);
    SB();
    MMA("", acc[2][6], bg[6], a[1][4], a[1][5]);
    SB();
    asm volatile("s_waitcnt vmcnt(5)" : "+v"(a[1][6]), "+v"(a[1][7]) :: "memory");
    am = min(am, MINBLK(1, 3));
    asm volatile("" : "+v"(am));
    SB();
    ADD4(1, 0, 2);
    SB();
    PUT(2, 5);
    SB();
    MMA("", acc[2][7], bf[7], a[0][4], a[0][5]);
    SB();
    SPIN(LDA(a[0][6], ao2[0], 2048); LDA(a[0][7], ao2[0], 3072); LDA(a[0][8], ao4[0], 0); LDA(a[0][9], ao4[0], 1024); LDA(a[1][6], ao2[1], 2048); LDA(a[1][7], ao2[1], 3072); LDA(a[1][8], ao4[1], 0); LDA(a[1][9], ao4[1], 1024); asm volatile("s_waitcnt vmcnt(0)" : "+v"(a[0][6]), "+v"(a[0][7]), "+v"(a[0][8]), "+v"(a[0][9]), "+v"(a[1][6]), "+v"(a[1][7]), "+v"(a[1][8]), "+v"(a[1][9]) :: "memory"); am = min(MINBLK(0, 3), MINBLK(1, 3)););
    SB();
    MUL4(1);
    SB();
    OUT(1, 1);
    SB();
    MMA("", acc[2][7], bg[7], a[1][4], a[1][5]);
    SB();
    PUT(2, 6);
    SB();
    MMA(NOP, acc[3][0], bf[0], a[0][6], a[0][7]);
    SB();
    MMA(NOP, acc[3][0], bg[0], a[1][6], a[1][7]);
    SB();
    PUT(2, 7);
    SB();
    MMA("", acc[3][1], bf[1], a[0][6], a[0][7]);
    SB();
    MMA("", acc[3][1], bg[1], a[1][6], a[1][7]);
    SB();
    PUT(3, 0);
    SB();
    MMA("", acc[3][2], bf[2], a[0][6], a[0][7]);
    SB();
    BAR(1);
    SB();
    GET(2, 0, 0);
    GET(2, 0, 1);
    SB();
    GET(2, 0, 2);
    GET(2, 0, 3);
    SB();
    MMA("", acc[3][2], bg[2], a[1][6], a[1][7]);
    SB();
    PUT(3, 1);
    SB();
    MMA("", acc[3][3], bf[3], a[0][6], a[0][7]);
    SB();
    GET(2, 1, 0);
    GET(2, 1, 1);
    SB();
    GET(2, 1, 2);
    GET(2, 1, 3);
    SB();
    MMA("", acc[3][3], bg[3], a[1][6], a[1][7]);
    SB();
    PUT(3, 2);
    SB();
    MMA("", acc[3][4], bf[4], a[0][6], a[0][7]);
    SB();
    asm volatile("s_waitcnt lgkmcnt(8)" : "+v"(gv[0][0]), "+v"(gv[0][1]) :: "memory");
    ADD4(0, 0, 1);
    SB();
    asm volatile("s_waitcnt lgkmcnt(6)" : "+v"(gv[0][2]), "+v"(gv[0][3]) :: "memory");
    ADD4(0, 2, 3);
    SB();
    MMA("", acc[3][4], bg[4], a[1][6], a[1][7]);
    SB();
    ADD4(0, 0, 2);
    SB();
    PUT(3, 3);
    SB();
    MMA("", acc[3][5], bf[5], a[0][6], a[0][7]);
    SB();
    MUL4(0);
    SB();
    OUT(2, 0);
    SB();
    MMA("", acc[3][5], bg[5], a[1][6], a[1][7]);
    SB();
    asm volatile("" : "+v"(a[0][8]), "+v"(a[0][9]) :: "memory");
    am = (MINBLK(0, 4) | no5);
    asm volatile("" : "+v"(am));
    SB();
    PUT(3, 4);
    SB();
    MMA("", acc[3][6], bf[6], a[0][6], a[0][7]);
    SB();
    asm volatile("s_waitcnt lgkmcnt(5)" : "+v"(gv[1][0]), "+v"(gv[1][1]) :: "memory");
    ADD4(1, 0, 1);
    SB();
    asm volatile("s_waitcnt lgkmcnt(3)" : "+v"(gv[1][2]), "+v"(gv[1][3]) :: "memory");
    ADD4(1, 2, 3);
    SB();
    MMA("", acc[3][6], bg[6], a[1][6], a[1][7]);
    SB();
    asm volatile("s_waitcnt vmcnt(4)" : "+v"(a[1][8]), "+v"(a[1][9]) :: "memory");
    am = min(am, (MINBLK(1, 4) | no5));
    asm volatile("" : "+v"(am));
    SB();
    ADD4(1, 0, 2);
    SB();
    PUT(3, 5);
    SB();
    MMA("", acc[3][7], bf[7], a[0][6], a[0][7]);
    SB();
    SPIN(LDA(a[0][8], ao4[0], 0); LDA(a[0][9], ao4[0], 1024); LDA(a[1][8], ao4[1], 0); LDA(a[1][9], ao4[1], 1024); asm volatile("s_waitcnt vmcnt(0)" : "+v"(a[0][8]), "+v"(a[0][9]), "+v"(a[1][8]), "+v"(a[1][9]) :: "memory"); am = min((MINBLK(0, 4) | no5), (MINBLK(1, 4) | no5)););
    SB();
    MUL4(1);
    SB();
    OUT(2, 1);
    SB();
    MMA("", acc[3][7], bg[7], a[1][6], a[1][7]);
    SB();
    PUT(3, 6);
    SB();
    MMA(NOP, acc[4][0], bf[0], a[0][8], a[0][9]);
    SB();
    MMA(NOP, acc[4][0], bg[0], a[1][8], a[1][9]);
    SB();
    PUT(3, 7);
    SB();
    MMA("", acc[4][1], bf[1], a[0][8], a[0][9]);
    SB();
    MMA("", acc[4][1], bg[1], a[1][8], a[1][9]);
    SB();
    PUT(4, 0);
    SB();
    MMA("", acc[4][2], bf[2], a[0][8], a[0][9]);
    SB();
    POISON(3);
    BAR(1);
    SB();
    GET(3, 0, 0);
    GET(3, 0, 1);
    SB();
    GET(3, 0, 2);
    GET(3, 0, 3);
    SB();
    MMA("", acc[4][2], bg[2], a[1][8], a[1][9]);
    SB();
    PUT(4, 1);
    SB();
    MMA("", acc[4][3], bf[3], a[0][8], a[0][9]);
    SB();
    GET(3, 1, 0);
    GET(3, 1, 1);
    SB();
    GET(3, 1, 2);
    GET(3, 1, 3);
    SB();
    MMA("", acc[4][3], bg[3], a[1][8], a[1][9]);
    SB();
    PUT(4, 2);
    SB();
    MMA("", acc[4][4], bf[4], a[0][8], a[0][9]);
    SB();
    asm volatile("s_waitcnt lgkmcnt(8)" : "+v"(gv[0][0]), "+v"(gv[0][1]) :: "memory");
    ADD4(0, 0, 1);
    SB();
    asm volatile("s_waitcnt lgkmcnt(6)" : "+v"(gv[0][2]), "+v"(gv[0][3]) :: "memory");
    ADD4(0, 2, 3);
    SB();
    MMA("", acc[4][4], bg[4], a[1][8], a[1][9]);
    SB();
    ADD4(0, 0, 2);
    SB();
    PUT(4, 3);
    SB();
    MMA("", acc[4][5], bf[5], a[0][8], a[0][9]);
    SB();
    MUL4(0);
    SB();
    OUT(3, 0);
    SB();
    MMA("", acc[4][5], bg[5], a[1][8], a[1][9]);
    SB();
    PUT(4, 4);
    SB();
    MMA("", acc[4][6], bf[6], a[0][8], a[0][9]);
    SB();
    asm volatile("s_waitcnt lgkmcnt(5)" : "+v"(gv[1][0]), "+v"(gv[1][1]) :: "memory");
    ADD4(1, 0, 1);
    SB();
    asm volatile("s_waitcnt lgkmcnt(3)" : "+v"(gv[1][2]), "+v"(gv[1][3]) :: "memory");
    ADD4(1, 2, 3);
    SB();
    MMA("", acc[4][6], bg[6], a[1][8], a[1][9]);
    SB();
    ADD4(1, 0, 2);
    SB();
    PUT(4, 5);
    SB();
    MMA("", acc[4][7], bf[7], a[0][8], a[0][9]);
    SB();
    MUL4(1);
    SB();
    OUT(3, 1);
    SB();
    MMA("", acc[4][7], bg[7], a[1][8], a[1][9]);
    SB();
    PUT(4, 6);
    SB();
    STAMP(10);
    asm volatile("s_nop 11" ::: "memory");
    PUT(4, 7);
    BAR(0);
    GET(4, 0, 0);
    GET(4, 0, 1);
    GET(4, 0, 2);
    GET(4, 0, 3);
    GET(4, 1, 0);
    GET(4, 1, 1);
    GET(4, 1, 2);
    GET(4, 1, 3);
    asm volatile("s_waitcnt lgkmcnt(6)" : "+v"(gv[0][0]), "+v"(gv[0][1]) :: "memory");
    ADD4(0, 0, 1);
    asm volatile("s_waitcnt lgkmcnt(4)" : "+v"(gv[0][2]), "+v"(gv[0][3]) :: "memory");
    ADD4(0, 2, 3);
    ADD4(0, 0, 2);
    MUL4(0);
    OUT(4, 0);
    asm volatile("s_waitcnt lgkmcnt(2)" : "+v"(gv[1][0]), "+v"(gv[1][1]) :: "memory");
    ADD4(1, 0, 1);
    asm volatile("s_waitcnt lgkmcnt(0)" : "+v"(gv[1][2]), "+v"(gv[1][3]) :: "memory");
    ADD4(1, 2, 3);
    ADD4(1, 0, 2);
    MUL4(1);
    OUT(4, 1);
    STAMP(11);
    STAMP(12);
    STAMP(13);
    STAMP_END();
  }
}

extern "C" void kernel_launch(int rank, int nranks, ncclComm_t comm, hipStream_t stream,
                              void* const* d_in, const int* in_sizes, int n_in,
                              void* d_out, int out_size, const int* in_dtypes, int out_dtype,
                              void* const* d_out_all, void* const* d_ws_all, size_t ws_bytes) {
  (void)comm; (void)in_dtypes; (void)out_dtype; (void)d_out_all;
  if (nranks != NR || (unsigned)rank >= (unsigned)NR || n_in != 4 || in_sizes[0] != KD * MS || in_sizes[1] != KD * ND ||
      out_size != MS * ND || ws_bytes < (size_t)WS_NEED) {
    fprintf(stderr, "a2a_gemm: unexpected problem shape\n");
    return;
  }
  static bool init = false;
  if (!init) {
    hipError_t e = hipFuncSetAttribute((const void*)a2a_gemm, hipFuncAttributeMaxDynamicSharedMemorySize, LDS_BYTES);
    if (e != hipSuccess) { fprintf(stderr, "hipFuncSetAttribute: %s\n", hipGetErrorString(e)); return; }
    init = true;
  }
  KArgs a;
  a.x = (const uint8_t*)d_in[0]; a.w = (const uint8_t*)d_in[1]; a.sx = (const float*)d_in[2]; a.sw = (const float*)d_in[3];
  a.out = (float*)d_out; a.rank = rank; a.own = (uint8_t*)d_ws_all[rank];
  for (int i = 0; i < NR; ++i) a.ws[i] = (uint8_t*)d_ws_all[(rank + 1 + i) % NR];
  hipLaunchKernelGGL(a2a_gemm, dim3(256), dim3(256), LDS_BYTES, stream, a);
}
```
